```python
import numpy as np

kernel_name = "gpu_gemm_rs_m4096_k4096_n16384_w8a8_dqsilu"


ABSMAX_RTOL = 0.001

COMPUTE_DTYPE = "int8"
EPILOGUE = "dequant_silu"
IN_QUANT = "int8"
OUT_QUANT = None
IN_DTYPES = {"x": "int8", "w_mat": "int8",
             "scale_x": "float32", "scale_w": "float32"}
WORLD_SIZE = 8

def setup_inputs():

    x = np.random.randint(-127, 128, size=(4096, 4096)).astype(np.int8)
    w_mat = np.random.randint(-127, 128, size=(4096, 16384)).astype(np.int8)
    scale_x = (0.02 * (1.0 + np.random.rand(1))).astype(np.float32)
    scale_w = (0.02 * (1.0 + np.random.rand(1))).astype(np.float32)
    return {"x": x, "w_mat": w_mat, "scale_x": scale_x, "scale_w": scale_w}

def reference(x, w_mat, scale_x, scale_w):
    acc = x.astype(np.float64) @ w_mat.astype(np.float64)
    y = (acc.astype(np.float32) * (scale_x * scale_w)).astype(np.float32)
    return (y / (1.0 + np.exp(-np.clip(y, -60.0, 60.0)))).astype(np.float32)
```

```cpp
#include <hip/hip_runtime.h>
#include <rccl/rccl.h>
#include <cstdio>
#include <cstdint>

#define HIP_CHECK(expr) do { hipError_t _e = (expr); if (_e != hipSuccess) { \
    fprintf(stderr, "HIP error %s at %s:%d\n", hipGetErrorString(_e), __FILE__, __LINE__); \
    return; } } while (0)

namespace {

typedef int      i32x4  __attribute__((ext_vector_type(4)));
typedef int      i32x16 __attribute__((ext_vector_type(16)));
typedef unsigned u32x4  __attribute__((ext_vector_type(4)));
typedef unsigned u32x2  __attribute__((ext_vector_type(2)));
typedef float    f32x4  __attribute__((ext_vector_type(4)));

constexpr int R = 8;
constexpr int GN = 16384;
constexpr int KS = 512;
constexpr int NR = 2048;
constexpr int NWG = 256, NTHR = 512;
#define T_GATE 12

constexpr unsigned XS_BYTES = 16u << 20;
constexpr unsigned WS_BYTES = 8u << 20;
constexpr unsigned BUF_BYTES = XS_BYTES + WS_BYTES;
constexpr unsigned CTL = 2 * BUF_BYTES;
constexpr unsigned XFLAG = CTL;
constexpr unsigned WFLAG = XFLAG + 16 * 128 * 4;
constexpr unsigned DONE = WFLAG + 16 * 64 * 4;
constexpr unsigned CNT = DONE + 1024;
constexpr unsigned TMO = CNT + 1024;
constexpr unsigned DBG = TMO + 1024;
constexpr unsigned WS_USED = DBG + 2 * 256 * 16 * 8;

constexpr int STAGE = 24576, NSLOT = 6, DEPTH = 5, NSTAGE = 64;
constexpr unsigned long long DEADLINE = 50000000ull;
constexpr int AUX_SYS = 17;
#define AUX_LD 17

struct Params {
  const int8_t* x; const int8_t* w; const float* sx; const float* sw;
  float* out[R]; char* ws[R];
  int rank;
};

#define WAIT_VM(n) asm volatile("s_waitcnt vmcnt(" #n ")" ::: "memory")
#define WAIT_VM_(n) WAIT_VM(n)
#define IN_WIN 1
#define OUT_WIN 0
#define OUT_HALF
#define IN_THROTTLE() WAIT_VM_(IN_WIN)
#define OUT_THROTTLE() WAIT_VM_(OUT_WIN)
#define RAW_BARRIER() do { asm volatile("s_waitcnt lgkmcnt(0)" ::: "memory"); __builtin_amdgcn_s_barrier(); asm volatile("" ::: "memory"); } while (0)

#define STAMP(i) do {} while (0)
#define STAMPW(i) do {} while (0)

__device__ __forceinline__ __amdgpu_buffer_rsrc_t make_rsrc(const void* p, unsigned bytes) {
  return __builtin_amdgcn_make_buffer_rsrc((void*)p, 0, bytes, 0x00020000);
}
__device__ __forceinline__ unsigned ld_sys(const unsigned* p) {
  return __hip_atomic_load(p, __ATOMIC_RELAXED, __HIP_MEMORY_SCOPE_SYSTEM);
}
__device__ __forceinline__ void st_sys(unsigned* p, unsigned v) {
  __hip_atomic_store(p, v, __ATOMIC_RELAXED, __HIP_MEMORY_SCOPE_SYSTEM);
}
__device__ __forceinline__ bool give_up(unsigned* tmo, unsigned call, unsigned long long t0) {
  unsigned t = __hip_atomic_load(tmo, __ATOMIC_RELAXED, __HIP_MEMORY_SCOPE_AGENT);
  t = __builtin_amdgcn_readfirstlane(t);
  if (t == call || __builtin_amdgcn_s_memrealtime() - t0 > DEADLINE) {
    __hip_atomic_store(tmo, call, __ATOMIC_RELAXED, __HIP_MEMORY_SCOPE_AGENT);
    return true;
  }
  return false;
}

__device__ __forceinline__ void transpose4(unsigned d0, unsigned d1, unsigned d2, unsigned d3,
                                           unsigned& o0, unsigned& o1, unsigned& o2, unsigned& o3) {
  unsigned t0 = __builtin_amdgcn_perm(d1, d0, 0x05010400u);
  unsigned t1 = __builtin_amdgcn_perm(d1, d0, 0x07030602u);
  unsigned t2 = __builtin_amdgcn_perm(d3, d2, 0x05010400u);
  unsigned t3 = __builtin_amdgcn_perm(d3, d2, 0x07030602u);
  o0 = __builtin_amdgcn_perm(t2, t0, 0x05040100u);
  o1 = __builtin_amdgcn_perm(t2, t0, 0x07060302u);
  o2 = __builtin_amdgcn_perm(t3, t1, 0x05040100u);
  o3 = __builtin_amdgcn_perm(t3, t1, 0x07060302u);
}

__device__ __forceinline__ float silu_dq(int acc, float s) {
  float y = (float)acc * s;
  float c = __builtin_fminf(__builtin_fmaxf(y, -60.0f), 60.0f);
  float e = __expf(-c);
  return y * __builtin_amdgcn_rcpf(1.0f + e);
}

__global__ void __launch_bounds__(NTHR, 2) fused_kernel(Params p) {
  __shared__ __attribute__((aligned(16))) char lds[STAGE * NSLOT];
  const int lane = threadIdx.x & 63;
  const int wave = __builtin_amdgcn_readfirstlane(threadIdx.x >> 6);
  const int bid = blockIdx.x;
  const int rank = p.rank;
  char* const myws = p.ws[rank];
  unsigned* const tmo = (unsigned*)(myws + TMO);
  unsigned* const cnt = (unsigned*)(myws + CNT) + bid;
  const unsigned cnt_v = __hip_atomic_load(cnt, __ATOMIC_RELAXED, __HIP_MEMORY_SCOPE_AGENT);
  unsigned call = 0, par = 0;
  auto get_call = [&]() {
    asm volatile("" ::: "memory");
    call = __builtin_amdgcn_readfirstlane(cnt_v) + 1u;
    par = (call & 1u) * BUF_BYTES;
  };
  const int h = lane >> 5, c31 = lane & 31;

  int rb, cb;
  const int xcd = bid & 7, wi = bid >> 3;
  {
    cb = xcd * 2 + (wi & 1);
    const int j = wi >> 1;
    if (j < 2) rb = rank * 2 + j;
    else if (j < 9) rb = ((rank + j - 1) & 7) * 2 + 1;
    else rb = ((rank + j - 8) & 7) * 2;
  }

  if (wi < 16 && wave < 2) {
    const int dst = (2 * wi + wave) & 7;
    char* const dws = p.ws[dst];
    const __amdgpu_buffer_rsrc_t drs = make_rsrc(dws, WS_USED);
    const __amdgpu_buffer_rsrc_t xrs = make_rsrc(p.x, 4096 * KS);
    const __amdgpu_buffer_rsrc_t wrs = make_rsrc(p.w, KS * GN);
    const int g = xcd * 4 + (wi >> 2);
    unsigned pend = 0;
    unsigned d[32];
    u32x4 v[8];
    const unsigned wvoff = (unsigned)(16 * h) * GN + 4 * c31;
    const unsigned xvoff = c31 * KS + 16 * h;
    auto load_w = [&](int kp) {
      const unsigned soff = (unsigned)(64 * kp) * GN + dst * NR + (g & 15) * 128;
#pragma unroll
      for (int i = 0; i < 32; ++i)
        d[i] = __builtin_amdgcn_raw_buffer_load_b32(wrs, wvoff, soff + ((i >> 4) * 32 + (i & 15)) * GN, 0);
    };
    auto load_x = [&](int trb, int tt) {
      const unsigned soff = (unsigned)(trb * 256) * KS + tt * 32;
#pragma unroll
      for (int rg = 0; rg < 8; ++rg) v[rg] = __builtin_amdgcn_raw_buffer_load_b128(xrs, xvoff, soff + rg * 32 * KS, 0);
    };
    auto publish = [&]() { if (pend && lane == 0) st_sys((unsigned*)(dws + pend), call); };
    auto store_w = [&](int kp) {
      WAIT_VM(0);
#pragma unroll
      for (int i = 0; i < 32; i += 8)
        asm volatile("" : "+v"(d[i]), "+v"(d[i + 1]), "+v"(d[i + 2]), "+v"(d[i + 3]), "+v"(d[i + 4]), "+v"(d[i + 5]), "+v"(d[i + 6]), "+v"(d[i + 7]) :: "memory");
      publish();
      const unsigned off = par + XS_BYTES + (unsigned)(((g & 15) * 8 + rank) * 16 + 2 * kp) * 4096 + lane * 16;
      pend = WFLAG + (((g & 15) * 8 + rank) * 8 + kp) * 4;
#pragma unroll
      for (int kk = 0; kk < 2; ++kk) {
        unsigned t[4][4];
#pragma unroll
        for (int q = 0; q < 4; ++q)
          transpose4(d[kk * 16 + 4 * q], d[kk * 16 + 4 * q + 1], d[kk * 16 + 4 * q + 2], d[kk * 16 + 4 * q + 3], t[0][q], t[1][q], t[2][q], t[3][q]);
#pragma unroll
        for (int b = 0; b < 4; ++b) {
          __builtin_amdgcn_raw_buffer_store_b128((u32x4){t[b][0], t[b][1], t[b][2], t[b][3]}, drs, off + (kk * 4 + b) * 1024, 0, AUX_SYS);
          IN_THROTTLE();
        }
      }
      asm volatile("" ::: "memory");
    };
    auto store_x = [&](int trb, int tt) {
      WAIT_VM(0);
      asm volatile("" : "+v"(v[0]), "+v"(v[1]), "+v"(v[2]), "+v"(v[3]), "+v"(v[4]), "+v"(v[5]), "+v"(v[6]), "+v"(v[7]) :: "memory");
      publish();
      const unsigned off = par + (unsigned)((trb * 8 + rank) * 16 + tt) * 8192 + lane * 16;
      pend = XFLAG + ((trb * 8 + rank) * 16 + tt) * 4;
#pragma unroll
      for (int rg = 0; rg < 8; ++rg) {
        __builtin_amdgcn_raw_buffer_store_b128(v[rg], drs, off + rg * 1024, 0, AUX_SYS);
        IN_THROTTLE();
      }
      asm volatile("" ::: "memory");
    };
    const int rbA = 2 * (g & 7), t0 = g >> 3;
    load_w(g >> 4);
    get_call();
#pragma unroll 1
    for (int pr = 0; pr < 4; ++pr) {
      store_w(2 * pr + (g >> 4));
      load_x(rbA, 4 * pr + t0);
      store_x(rbA, 4 * pr + t0);
      if (pr < 3) load_w(2 * pr + 2 + (g >> 4));
    }
#pragma unroll 1
    for (int pr = 0; pr < 4; ++pr) {
      load_x(rbA + 1, 4 * pr + t0);
      store_x(rbA + 1, 4 * pr + t0);
    }
    WAIT_VM(0);
    publish();
    STAMPW(8 + wave);
  } else {
    get_call();
  }

  const __amdgpu_buffer_rsrc_t mrs = make_rsrc(myws, WS_USED);
  auto stage_id = [&](int q) {
    return ((rank + q) & 7) * 8 + (q >> 3);
  };
  const int my_id = stage_id(lane);
  const unsigned* const xf = (const unsigned*)(myws + XFLAG) + rb * 128 + my_id * 2;
  const unsigned* const wf = (const unsigned*)(myws + WFLAG) + cb * 64 + my_id;
  int ready = 0;
  auto ensure = [&](int q) {
    if (ready > q) return;
    const unsigned long long t0 = __builtin_amdgcn_s_memrealtime();
    for (;;) {
      u32x2 fx; unsigned fw;
      asm volatile("global_load_dwordx2 %0, %2, off sc0 sc1\n\t"
                   "global_load_dword %1, %3, off sc0 sc1\n\t"
                   "s_waitcnt vmcnt(0)"
                   : "=&v"(fx), "=&v"(fw) : "v"(xf), "v"(wf) : "memory");
      const unsigned long long m = __ballot(fx.x >= call && fx.y >= call && fw >= call);
      ready = (~m == 0ull) ? 64 : __builtin_ctzll(~m);
      if (ready > q) {
        return;
      }
      if (give_up(tmo, call, t0)) { ready = 64; return; }
      __builtin_amdgcn_s_sleep(4);
    }
  };
  const unsigned a_base = par + (unsigned)rb * (1u << 20) + wave * 2048;
  const unsigned b_base = par + XS_BYTES + (unsigned)cb * (512u << 10) + wave * 1024;
  auto issue = [&](int q, int slot) {
    const int id = stage_id(q);
    char* l = lds + slot * STAGE;
    auto* l0 = (__attribute__((address_space(3))) void*)(l + wave * 2048);
    auto* l1 = (__attribute__((address_space(3))) void*)(l + wave * 2048 + 1024);
    auto* l2 = (__attribute__((address_space(3))) void*)(l + 16384 + wave * 1024);
    __builtin_amdgcn_raw_ptr_buffer_load_lds(mrs, l0, 16, lane * 16, a_base + id * 16384, 0, AUX_LD);
    __builtin_amdgcn_raw_ptr_buffer_load_lds(mrs, l1, 16, lane * 16, a_base + id * 16384 + 1024, 0, AUX_LD);
    __builtin_amdgcn_raw_ptr_buffer_load_lds(mrs, l2, 16, lane * 16, b_base + id * 8192, 0, AUX_LD);
  };
  i32x16 acc[4];
#pragma unroll
  for (int b = 0; b < 4; ++b)
#pragma unroll
    for (int i = 0; i < 16; ++i) acc[b][i] = 0;
  auto compute = [&](int slot) {
    const char* l = lds + slot * STAGE + lane * 16;
#pragma unroll
    for (int ks = 0; ks < 2; ++ks) {
      const i32x4 a = *(const i32x4*)(l + ks * 8192 + wave * 1024);
      i32x4 bf[4];
#pragma unroll
      for (int b = 0; b < 4; ++b) bf[b] = *(const i32x4*)(l + 16384 + ks * 4096 + b * 1024);
#pragma unroll
      for (int b = 0; b < 4; ++b) acc[b] = __builtin_amdgcn_mfma_i32_32x32x32_i8(a, bf[b], acc[b], 0, 0, 0);
    }
  };

  if (wave == 0) ensure(DEPTH - 1);
  RAW_BARRIER();
  STAMP(1);
#pragma unroll
  for (int q = 0; q < DEPTH; ++q) issue(q, q);
  int slot = 0;
  for (int i = 0; i < NSTAGE - DEPTH; ++i) {
    if (wave == 0) ensure(i + DEPTH);
    WAIT_VM(12);
    RAW_BARRIER();
    issue(i + DEPTH, slot == 0 ? NSLOT - 1 : slot - 1);
    compute(slot);
    slot = (slot == NSLOT - 1) ? 0 : slot + 1;
  }
  STAMP(2);
#define TAIL(n) do { WAIT_VM(n); RAW_BARRIER(); compute(slot); slot = (slot == NSLOT - 1) ? 0 : slot + 1; } while (0)
  TAIL(12); TAIL(9); TAIL(6); TAIL(3); TAIL(0);
#undef TAIL
  STAMP(3);

  if (T_GATE > 0 && wi >= 18) {
    if (wave == 0) {
      const unsigned* gf = (const unsigned*)(myws + XFLAG) + (rb | 1) * 128 + lane * 2;
      const bool dont_care = ((lane & 7) * 2 + 1) >= T_GATE;
      const unsigned long long t0 = __builtin_amdgcn_s_memrealtime();
      for (;;) {
        u32x2 f;
        asm volatile("global_load_dwordx2 %0, %1, off sc0 sc1\n\ts_waitcnt vmcnt(0)" : "=&v"(f) : "v"(gf) : "memory");
        if (__all(dont_care || (f.x >= call && f.y >= call))) break;
        if (give_up(tmo, call, t0)) break;
        __builtin_amdgcn_s_sleep(8);
      }
    }
    RAW_BARRIER();
  }
  STAMP(6);
  const int owner = rb >> 1;
  {
    const bool bad = __builtin_amdgcn_readfirstlane(
        __hip_atomic_load(tmo, __ATOMIC_RELAXED, __HIP_MEMORY_SCOPE_AGENT)) == call;
    const float s = bad ? __builtin_nanf("") : p.sx[0] * p.sw[0];
    const __amdgpu_buffer_rsrc_t ors = make_rsrc(p.out[owner], 512u * GN * 4u);
    const unsigned voff = (unsigned)(4 * h) * (GN * 4) + c31 * 16;
    const unsigned soff = (unsigned)((rb & 1) * 256 + wave * 32) * (GN * 4) + (rank * NR + cb * 128) * 4;
    if (wave >= 4) RAW_BARRIER();
#pragma unroll
    for (int i = 0; i < 16; ++i) {
      f32x4 v;
#pragma unroll
      for (int b = 0; b < 4; ++b) v[b] = silu_dq(acc[b][i], s);
      __builtin_amdgcn_raw_buffer_store_b128(__builtin_bit_cast(u32x4, v), ors,
                                             voff + soff + ((i & 3) + 8 * (i >> 2)) * (GN * 4), 0, AUX_SYS);
      OUT_THROTTLE();
    }
    if (wave < 4) RAW_BARRIER();
  }
  WAIT_VM(0);
  RAW_BARRIER();
  STAMP(4);
  if (wave == 0) {
    if (lane == 0) {
      if (owner != rank) st_sys((unsigned*)(p.ws[owner] + DONE) + rank * 32 + (rb & 1) * 16 + cb, call);
      __hip_atomic_store(cnt, call, __ATOMIC_RELAXED, __HIP_MEMORY_SCOPE_AGENT);
    }
    if (bid == 0) {
      const unsigned* df = (const unsigned*)(myws + DONE) + lane * 4;
      const bool mine = (lane >> 3) == rank;
      const unsigned long long t0 = __builtin_amdgcn_s_memrealtime();
      for (;;) {
        u32x4 f;
        asm volatile("global_load_dwordx4 %0, %1, off sc0 sc1\n\ts_waitcnt vmcnt(0)" : "=&v"(f) : "v"(df) : "memory");
        const bool ok = mine || (f.x >= call && f.y >= call && f.z >= call && f.w >= call);
        if (__all(ok)) break;
        if (give_up(tmo, call, t0)) {
          if (lane == 0) p.out[rank][(size_t)((rb & 1) * 256) * GN + rank * NR + cb * 128] = __builtin_nanf("");
          break;
        }
        __builtin_amdgcn_s_sleep(2);
      }
      STAMP(5);
    }
  }
}

}

extern "C" void kernel_launch(int rank, int nranks, ncclComm_t comm,
                              hipStream_t stream,
                              void* const* d_in, const int* in_sizes, int n_in,
                              void* d_out, int out_size,
                              const int* in_dtypes, int out_dtype,
                              void* const* d_out_all,
                              void* const* d_ws_all, size_t ws_bytes) {
  (void)comm; (void)in_sizes; (void)n_in; (void)d_out; (void)out_size; (void)in_dtypes; (void)out_dtype;
  if (nranks != R || (unsigned)rank >= (unsigned)R || ws_bytes < WS_USED) {
    fprintf(stderr, "unsupported configuration\n");
    return;
  }
  Params p;
  p.x = (const int8_t*)d_in[0];
  p.w = (const int8_t*)d_in[1];
  p.sx = (const float*)d_in[2];
  p.sw = (const float*)d_in[3];
  for (int i = 0; i < R; ++i) { p.out[i] = (float*)d_out_all[i]; p.ws[i] = (char*)d_ws_all[i]; }
  p.rank = rank;
  fused_kernel<<<NWG, NTHR, 0, stream>>>(p);
  HIP_CHECK(hipGetLastError());
}
```

```cpp
#include <hip/hip_runtime.h>
#include <rccl/rccl.h>
#include <cstdio>
#include <cstdint>

#define HIP_CHECK(expr) do { hipError_t _e = (expr); if (_e != hipSuccess) { \
    fprintf(stderr, "HIP error %s at %s:%d\n", hipGetErrorString(_e), __FILE__, __LINE__); \
    return; } } while (0)

#define NWO 24
#define NWP 24
#define POLL_SLEEP 4
#define O_WIN 0
#define L_WIN 1
#define O_SIG 14
#define LG1 5

namespace {

typedef int      i32x4  __attribute__((ext_vector_type(4)));
typedef int      i32x16 __attribute__((ext_vector_type(16)));
typedef unsigned u32x4  __attribute__((ext_vector_type(4)));
typedef unsigned u32x2  __attribute__((ext_vector_type(2)));
typedef float    f32x4  __attribute__((ext_vector_type(4)));

constexpr int R = 8;
constexpr int GN = 16384;
constexpr int KS = 512;
constexpr int NWG = 256, NTHR = 512;

constexpr unsigned XS_BYTES = 8u << 20;
constexpr unsigned WS_BYTES = 16u << 20;
constexpr unsigned BUF_BYTES = XS_BYTES + WS_BYTES;
constexpr unsigned CTL = 2 * BUF_BYTES;
constexpr unsigned XFLAG = CTL;
constexpr unsigned WFLAG = XFLAG + 8 * 128 * 4;
constexpr unsigned DONE = WFLAG + 32 * 64 * 4;
constexpr unsigned CNT = DONE + 1024;
constexpr unsigned TMO = CNT + 1024;
constexpr unsigned QUE = TMO + 1024;
constexpr unsigned CHAIN = QUE + 1024;
constexpr unsigned DBG = CHAIN + 48 * 128;
constexpr unsigned WS_USED = DBG + 2 * 256 * 16 * 8;

constexpr int STAGE = 24576, NSLOT = 6, NSTAGE = 64;
constexpr unsigned long long DEADLINE = 50000000ull;
constexpr int AUX_SYS = 17;

struct Params {
  const int8_t* x; const int8_t* w; const float* sx; const float* sw;
  float* out[R]; char* ws[R];
  int rank;
};

#define WAIT_VM(n) asm volatile("s_waitcnt vmcnt(" #n ")" ::: "memory")
#define WAIT_VM_(n) WAIT_VM(n)
#define RAW_BARRIER() do { asm volatile("s_waitcnt lgkmcnt(0)" ::: "memory"); __builtin_amdgcn_s_barrier(); asm volatile("" ::: "memory"); } while (0)

#define STAMP(i) do {} while (0)
#define STAMPW(i) do {} while (0)

__device__ __forceinline__ __amdgpu_buffer_rsrc_t make_rsrc(const void* p, unsigned bytes) {
  return __builtin_amdgcn_make_buffer_rsrc((void*)p, 0, bytes, 0x00020000);
}
__device__ __forceinline__ void st_sys(unsigned* p, unsigned v) {
  __hip_atomic_store(p, v, __ATOMIC_RELAXED, __HIP_MEMORY_SCOPE_SYSTEM);
}
__device__ __forceinline__ bool give_up(unsigned* tmo, unsigned call, unsigned long long t0) {
  unsigned t = __hip_atomic_load(tmo, __ATOMIC_RELAXED, __HIP_MEMORY_SCOPE_AGENT);
  t = __builtin_amdgcn_readfirstlane(t);
  if (t == call || __builtin_amdgcn_s_memrealtime() - t0 > DEADLINE) {
    __hip_atomic_store(tmo, call, __ATOMIC_RELAXED, __HIP_MEMORY_SCOPE_AGENT);
    return true;
  }
  return false;
}
__device__ __forceinline__ int xmask(int j) { return j + 1 + (j >= 3); }

__device__ __forceinline__ void transpose4(unsigned d0, unsigned d1, unsigned d2, unsigned d3,
                                           unsigned& o0, unsigned& o1, unsigned& o2, unsigned& o3) {
  unsigned t0 = __builtin_amdgcn_perm(d1, d0, 0x05010400u);
  unsigned t1 = __builtin_amdgcn_perm(d1, d0, 0x07030602u);
  unsigned t2 = __builtin_amdgcn_perm(d3, d2, 0x05010400u);
  unsigned t3 = __builtin_amdgcn_perm(d3, d2, 0x07030602u);
  o0 = __builtin_amdgcn_perm(t2, t0, 0x05040100u);
  o1 = __builtin_amdgcn_perm(t2, t0, 0x07060302u);
  o2 = __builtin_amdgcn_perm(t3, t1, 0x05040100u);
  o3 = __builtin_amdgcn_perm(t3, t1, 0x07060302u);
}

__device__ __forceinline__ float silu_dq(int acc, float s) {
  float y = (float)acc * s;
  float c = __builtin_amdgcn_fmed3f(y, -60.0f, 60.0f);
  float e = __builtin_amdgcn_exp2f(c * -1.44269504088896341f);
  return y * __builtin_amdgcn_rcpf(1.0f + e);
}

struct Task { int type, a, b, src; };

__global__ void __launch_bounds__(NTHR, 2) fused_kernel(Params p) {
  __shared__ __attribute__((aligned(16))) char lds[STAGE * NSLOT];
  const int lane = threadIdx.x & 63;
  const int wave = __builtin_amdgcn_readfirstlane(threadIdx.x >> 6);
  const int bid = blockIdx.x;
  const int rank = p.rank;
  char* const myws = p.ws[rank];
  unsigned* const tmo = (unsigned*)(myws + TMO);
  unsigned* const cnt = (unsigned*)(myws + CNT) + bid;
  const unsigned cnt_v = __hip_atomic_load(cnt, __ATOMIC_RELAXED, __HIP_MEMORY_SCOPE_AGENT);
  unsigned call = 0, par = 0;
  auto get_call = [&]() {
    asm volatile("" ::: "memory");
    call = __builtin_amdgcn_readfirstlane(cnt_v) + 1u;
    par = (call & 1u) * BUF_BYTES;
  };
  const int h = lane >> 5, c31 = lane & 31;
  const int xcd = bid & 7, wi = bid >> 3;
  const __amdgpu_buffer_rsrc_t mrs = make_rsrc(myws, WS_USED);

  int cw = -1, cn = 0, dst = rank, ntask = 0, cdj = -1;
  if (wi == 0) { cw = wave * 8 + xcd; cn = NWP; dst = rank ^ 4; ntask = 320 + 448; }
  else if (wi < 8 && wave < NWO / 8) {
    const int id = (wi - 1) * (NWO / 8) + wave, k = id / 7, dj = id - 7 * k;
    cw = k * 8 + xcd; cn = NWO;
    if (dj < 6) { dst = rank ^ xmask(dj); ntask = 320; cdj = dj; } else ntask = 320 + 64;
  }
  if (cw >= 0 && cw < cn) {
    char* const dws = p.ws[dst];
    const __amdgpu_buffer_rsrc_t drs = make_rsrc(dws, WS_USED);
    const int dhf = dst >> 2;
    auto decode = [&](int i) {
      Task t; t.src = rank;
      if (i < 224) {
        const int kq = i / 56, u = i - kq * 56;
        if (u < 24) { const int rq = u / 6; t.type = 1; t.a = 2 + u - 6 * rq; t.b = kq * 4 + rq; }
        else { t.type = 0; t.a = (u - 24) & 15; t.b = 2 * kq + ((u - 24) >> 4); }
      } else if (i < 288) { t.type = 0; t.a = 16 + ((i - 224) & 7); t.b = (i - 224) >> 3; }
      else if (i < 320) { t.type = 1; t.a = i & 1; t.b = (i - 288) >> 1; }
      else if (wi != 0) { t.type = 0; t.a = 24 + (i & 7); t.b = (i - 320) >> 3; }
      else {
        int j = i - 320, js = 0, gs = LG1;
        if (j >= LG1 * 56) { j -= LG1 * 56; js = LG1; gs = 8 - LG1; }
        const int kp = j / (gs * 7), u = j - kp * gs * 7, jb = u / 7, k = u - jb * 7;
        t.a = 24 + js + jb; t.b = kp;
        if (k == 0) t.type = 0; else { t.type = 2; t.src = rank ^ xmask(k - 1); }
      }
      return t;
    };
    unsigned pend = 0;
    bool given = false;
    char* const cl = lds + wave * 16384;
    const int r8 = lane >> 3, k8 = lane & 7;
    auto fetch = [&](const Task& t, int slot) {
      if (t.type == 3) return;
      const char* src; unsigned stride;
      const int cb = dhf ? 31 - t.a : t.a;
      if (t.type == 0) {
        src = (const char*)p.w + (size_t)(64 * t.b) * GN + (dst & 3) * 4096 + cb * 128; stride = GN;
      } else if (t.type == 1) {
        const int row = t.a < 2 ? 512 * dst + 256 * t.a : 512 * (dst ^ xmask(t.a - 2)) + 256 * dhf;
        src = (const char*)p.x + (size_t)(row + (t.b & 3) * 64) * KS + (t.b >> 2) * 128; stride = KS;
      } else {
        if (!given) {
          const unsigned* gf = (const unsigned*)(myws + WFLAG) + (dhf ? 0 : 24 * 64) + lane * 8;
          const bool skip = ((lane & 3) == (rank & 3));
          const unsigned long long t0 = __builtin_amdgcn_s_memrealtime();
          for (;;) {
            u32x4 f0, f1;
            asm volatile("global_load_dwordx4 %0, %2, off sc0 sc1\n\t"
                         "global_load_dwordx4 %1, %2, off offset:16 sc0 sc1\n\t"
                         "s_waitcnt vmcnt(0)" : "=&v"(f0), "=&v"(f1) : "v"(gf) : "memory");
            const unsigned m = min(min(min(f0.x, f0.y), min(f0.z, f0.w)), min(min(f1.x, f1.y), min(f1.z, f1.w)));
            if (__all(skip || m >= call)) break;
            if (give_up(tmo, call, t0)) break;
            __builtin_amdgcn_s_sleep(8);
          }
          given = true;
        }
        src = myws + par + XS_BYTES + (unsigned)((cb * 8 + t.src) * 16 + 2 * t.b) * 4096; stride = 128;
      }
      const __amdgpu_buffer_rsrc_t srs = make_rsrc(src, 63 * stride + 128);
      const unsigned voff = r8 * stride + ((t.type == 1 ? k8 ^ r8 : k8) * 16);
#pragma unroll
      for (int j = 0; j < 8; ++j)
        __builtin_amdgcn_raw_ptr_buffer_load_lds(srs, (__attribute__((address_space(3))) void*)(cl + slot * 8192 + j * 1024),
                                                 16, voff, j * 8 * stride, 0, AUX_SYS);
    };
    const unsigned qoff = lane == 0 ? QUE + dst * 128 : 0xfffffff0u;
    Task t = decode(cw);
    fetch(t, 0);
    Task tn = decode(cw + cn);
    get_call();
    int slot = 0;
    WAIT_VM(0);
#pragma unroll 1
    for (;;) {
      const char* l = cl + slot * 8192;
      const int cb = dhf ? 31 - t.a : t.a;
      unsigned off = par + XS_BYTES + (unsigned)((cb * 8 + t.src) * 16 + 2 * t.b) * 4096 + lane * 16;
      unsigned flag = WFLAG + ((cb * 8 + t.src) * 8 + t.b) * 4;
      int ticket;
      auto head = [&]() { WAIT_VM(0); if (pend && lane == 0) st_sys((unsigned*)(dws + pend), call); };
      auto mid = [&]() {
        asm volatile("" ::: "memory");
        ticket = __builtin_amdgcn_raw_ptr_buffer_atomic_add_i32(1, mrs, qoff, 0, 0);
        asm volatile("" ::: "memory");
        fetch(tn, slot ^ 1);
        asm volatile("" ::: "memory");
      };
      auto pace = [&]() { if (tn.type == 3) WAIT_VM(4); else WAIT_VM(12); };
      if (t.type == 0) {
        unsigned o[2][4][4];
#pragma unroll
        for (int kk = 0; kk < 2; ++kk)
#pragma unroll
          for (int q = 0; q < 4; ++q) {
            const unsigned* lp = (const unsigned*)(l + (kk * 32 + h * 16 + q * 4) * 128 + c31 * 4);
            transpose4(lp[0], lp[32], lp[64], lp[96], o[kk][0][q], o[kk][1][q], o[kk][2][q], o[kk][3][q]);
          }
#pragma unroll
        for (int b = 0; b < 4; ++b) asm volatile("" : "+v"(o[0][b][0]), "+v"(o[0][b][1]), "+v"(o[0][b][2]), "+v"(o[0][b][3]));
        head();
#pragma unroll
        for (int b = 0; b < 4; ++b)
          __builtin_amdgcn_raw_buffer_store_b128((u32x4){o[0][b][0], o[0][b][1], o[0][b][2], o[0][b][3]}, drs, off + b * 1024, 0, AUX_SYS);
        mid();
#pragma unroll
        for (int b = 0; b < 4; ++b) {
          pace();
          __builtin_amdgcn_raw_buffer_store_b128((u32x4){o[1][b][0], o[1][b][1], o[1][b][2], o[1][b][3]}, drs, off + (4 + b) * 1024, 0, AUX_SYS);
        }
      } else {
        unsigned sA = 1024, sB = 4096;
        if (t.type == 1) {
          off = par + (unsigned)((t.a * 8 + rank) * 16 + (t.b >> 2) * 4) * 8192 + (t.b & 3) * 2048 + lane * 16;
          flag = XFLAG + ((t.a * 8 + rank) * 16 + t.b) * 4;
          sA = 8192; sB = 1024;
        }
        u32x4 v[8];
#pragma unroll
        for (int j = 0; j < 8; ++j) {
          const int ks = j >> 1;
          const unsigned lo = t.type == 1 ? c31 * 128 + (((2 * ks + h) ^ (c31 & 7)) * 16) : ks * 1024 + lane * 16;
          v[j] = *(const u32x4*)(l + lo + (j & 1) * 4096);
        }
        asm volatile("" : "+v"(v[0]), "+v"(v[1]), "+v"(v[2]), "+v"(v[3]));
        head();
#pragma unroll
        for (int j = 0; j < 4; ++j)
          __builtin_amdgcn_raw_buffer_store_b128(v[j], drs, off + (j >> 1) * sA + (j & 1) * sB, 0, AUX_SYS);
        mid();
#pragma unroll
        for (int j = 4; j < 8; ++j) {
          pace();
          __builtin_amdgcn_raw_buffer_store_b128(v[j], drs, off + (j >> 1) * sA + (j & 1) * sB, 0, AUX_SYS);
        }
      }
      pend = flag;
      if (tn.type == 3) break;
      t = tn;
      slot ^= 1;
      WAIT_VM(4);
      const unsigned i = (unsigned)__builtin_amdgcn_readfirstlane(ticket) - (call - 1u) * (unsigned)ntask + 2u * (unsigned)cn;
      if (i < (unsigned)ntask) tn = decode((int)i); else tn.type = 3;
    }
    if (cdj >= 0 && lane < 8 && (lane * (NWO / 8)) >> 3 == cw >> 3)
      __hip_atomic_store((unsigned*)(myws + CHAIN + (xcd * 6 + cdj) * 128) + lane, call * 8u, __ATOMIC_RELAXED, __HIP_MEMORY_SCOPE_AGENT);
    WAIT_VM(0);
    if (lane == 0) st_sys((unsigned*)(dws + pend), call);
    STAMPW(8 + wave);
  } else {
    get_call();
  }

  const int njob = wi == 0 ? 0 : (wi == 1 ? 2 : 1);
#pragma unroll 1
  for (int job = 0; job < njob; ++job) {
    int lrb, jc;
    if (wi < 8) { lrb = wi == 1 ? job : (wi & 1); jc = wi >> 1; }
    else { jc = (wi - 8) / 6; lrb = 2 + (wi - 8) - 6 * jc; }
    const int cp = (wi >= 26) ? 24 + ((xcd + 3 * (lrb - 2)) & 7) : xcd + 8 * jc;
    const int cb = (rank >> 2) ? 31 - cp : cp;
    auto stage_id = [&](int q) { return ((rank + q) & 7) * 8 + (q >> 3); };
    const int my_id = stage_id(lane);
    const unsigned* const xf = (const unsigned*)(myws + XFLAG) + lrb * 128 + (my_id >> 1) * 4;
    const unsigned* const wf = (const unsigned*)(myws + WFLAG) + cb * 64 + my_id;
    int ready = 0;
    auto ensure = [&](int q) {
      if (ready > q) return;
      const unsigned long long t0 = __builtin_amdgcn_s_memrealtime();
      unsigned polls = 0;
      for (;;) {
        u32x4 fx; unsigned fw;
        asm volatile("global_load_dwordx4 %0, %2, off sc0 sc1\n\t"
                     "global_load_dword %1, %3, off sc0 sc1\n\t"
                     "s_waitcnt vmcnt(0)"
                     : "=&v"(fx), "=&v"(fw) : "v"(xf), "v"(wf) : "memory");
        const unsigned long long m = __ballot(min(min(min(fx.x, fx.y), min(fx.z, fx.w)), fw) >= call);
        ready = (~m == 0ull) ? 64 : __builtin_ctzll(~m);
        if (ready > q) return;
        if ((++polls & 15) == 0 && give_up(tmo, call, t0)) { ready = 64; return; }
        __builtin_amdgcn_s_sleep(POLL_SLEEP);
      }
    };
    const unsigned a_base = par + (unsigned)lrb * (1u << 20) + wave * 2048;
    const unsigned b_base = par + XS_BYTES + (unsigned)cb * (512u << 10) + wave * 1024;
    auto issue = [&](int q, int slot) {
      const int id = stage_id(q);
      char* l = lds + slot * STAGE;
      auto* l0 = (__attribute__((address_space(3))) void*)(l + wave * 2048);
      auto* l1 = (__attribute__((address_space(3))) void*)(l + wave * 2048 + 1024);
      auto* l2 = (__attribute__((address_space(3))) void*)(l + 16384 + wave * 1024);
      __builtin_amdgcn_raw_ptr_buffer_load_lds(mrs, l0, 16, lane * 16, a_base + id * 16384, 0, AUX_SYS);
      __builtin_amdgcn_raw_ptr_buffer_load_lds(mrs, l1, 16, lane * 16, a_base + id * 16384 + 1024, 0, AUX_SYS);
      __builtin_amdgcn_raw_ptr_buffer_load_lds(mrs, l2, 16, lane * 16, b_base + id * 8192, 0, AUX_SYS);
    };
    i32x16 acc[4];
#pragma unroll
    for (int b = 0; b < 4; ++b)
#pragma unroll
      for (int i = 0; i < 16; ++i) acc[b][i] = 0;
    i32x4 fa[2][2], fb[2][2][4];
    auto rd = [&](int buf, int slot) {
      const char* l = lds + slot * STAGE + lane * 16;
#pragma unroll
      for (int ks = 0; ks < 2; ++ks) {
        fa[buf][ks] = *(const i32x4*)(l + ks * 8192 + wave * 1024);
#pragma unroll
        for (int b = 0; b < 4; ++b) fb[buf][ks][b] = *(const i32x4*)(l + 16384 + ks * 4096 + b * 1024);
      }
    };
    auto mm = [&](int buf) {
#pragma unroll
      for (int ks = 0; ks < 2; ++ks)
#pragma unroll
        for (int b = 0; b < 4; ++b) acc[b] = __builtin_amdgcn_mfma_i32_32x32x32_i8(fa[buf][ks], fb[buf][ks][b], acc[b], 0, 0, 0);
    };
    auto nx = [&](int s) { return s == NSLOT - 1 ? 0 : s + 1; };

    if (wave == 0) ensure(NSLOT - 1);
    RAW_BARRIER();
    STAMP(1);
#pragma unroll
    for (int q = 0; q < NSLOT; ++q) issue(q, q);
    WAIT_VM(15);
    RAW_BARRIER();
    rd(0, 0);
    int s0 = 0;
#pragma unroll 1
    for (int i = 0; i < NSTAGE - NSLOT; i += 2) {
      if (wave == 0) ensure(i + NSLOT);
      WAIT_VM(12); RAW_BARRIER();
      issue(i + NSLOT, s0); rd(1, nx(s0)); mm(0);
      s0 = nx(s0);
      if (wave == 0) ensure(i + NSLOT + 1);
      WAIT_VM(12); RAW_BARRIER();
      issue(i + NSLOT + 1, s0); rd(0, nx(s0)); mm(1);
      s0 = nx(s0);
    }
    STAMP(2);
    WAIT_VM(12); RAW_BARRIER(); rd(1, nx(s0)); mm(0); s0 = nx(s0);
    WAIT_VM(9);  RAW_BARRIER(); rd(0, nx(s0)); mm(1); s0 = nx(s0);
    WAIT_VM(6);  RAW_BARRIER(); rd(1, nx(s0)); mm(0); s0 = nx(s0);
    WAIT_VM(3);  RAW_BARRIER(); rd(0, nx(s0)); mm(1); s0 = nx(s0);
    WAIT_VM(0);  RAW_BARRIER(); rd(1, nx(s0)); mm(0);
    mm(1);
    STAMP(3);

    const int owner = lrb < 2 ? rank : rank ^ xmask(lrb - 2);
    const int rowh = lrb < 2 ? lrb : rank >> 2;
    const float scale = p.sx[0] * p.sw[0];
    auto convert = [&](int i) {
#pragma unroll
      for (int b = 0; b < 4; ++b) acc[b][i] = __builtin_bit_cast(int, silu_dq(acc[b][i], scale));
    };
    bool pre = false;
    unsigned* const chain = (unsigned*)(myws + CHAIN + (xcd * 6 + lrb - 2) * 128) + wave;
    if (owner != rank) {
      const unsigned long long t0 = __builtin_amdgcn_s_memrealtime();
      for (unsigned polls = 1;; ++polls) {
        if (__builtin_amdgcn_readfirstlane(__hip_atomic_load(chain, __ATOMIC_RELAXED, __HIP_MEMORY_SCOPE_AGENT)) >= call * 8u + jc) break;
        if (!pre) {
#pragma unroll
          for (int i = 0; i < 16; ++i) convert(i);
          pre = true;
          continue;
        }
        if ((polls & 15) == 0 && give_up(tmo, call, t0)) break;
        __builtin_amdgcn_s_sleep(12);
      }
    }
    STAMP(7);
    {
      const bool bad = __builtin_amdgcn_readfirstlane(
          __hip_atomic_load(tmo, __ATOMIC_RELAXED, __HIP_MEMORY_SCOPE_AGENT)) == call;
      const int poison = bad ? 0x7fc00000 : 0;
      const __amdgpu_buffer_rsrc_t ors = make_rsrc(p.out[owner], 512u * GN * 4u);
      const unsigned voff = (unsigned)(rowh * 256 + wave * 32 + 4 * h) * (GN * 4) + ((rank & 3) * 4096 + cb * 128) * 4 + c31 * 16;
#pragma unroll
      for (int i = 0; i < 16; ++i) {
        if (!pre) convert(i);
        const u32x4 o = {(unsigned)(acc[0][i] | poison), (unsigned)(acc[1][i] | poison), (unsigned)(acc[2][i] | poison), (unsigned)(acc[3][i] | poison)};
        __builtin_amdgcn_raw_buffer_store_b128(o, ors, voff + ((i & 3) + 8 * (i >> 2)) * (GN * 4), 0, AUX_SYS);
        if (owner != rank) {
          if (i == O_SIG && lane == 0) __hip_atomic_store(chain, call * 8u + jc + 1u, __ATOMIC_RELAXED, __HIP_MEMORY_SCOPE_AGENT);
          if (jc < 3) WAIT_VM_(O_WIN); else WAIT_VM_(L_WIN);
        }
      }
    }
    WAIT_VM(0);
    RAW_BARRIER();
    STAMP(4);
    if (wave == 0 && lane == 0 && owner != rank) st_sys((unsigned*)(p.ws[owner] + DONE) + rank * 32 + cb, call);
  }

  if (wi == 0) RAW_BARRIER();
  if (wave == 0) {
    if (lane == 0) __hip_atomic_store(cnt, call, __ATOMIC_RELAXED, __HIP_MEMORY_SCOPE_AGENT);
    if (bid == 8) {
      const unsigned* df = (const unsigned*)(myws + DONE) + lane * 4;
      const bool none = ((lane >> 3) & 3) == (rank & 3);
      const unsigned long long t0 = __builtin_amdgcn_s_memrealtime();
      for (;;) {
        u32x4 f;
        asm volatile("global_load_dwordx4 %0, %1, off sc0 sc1\n\ts_waitcnt vmcnt(0)" : "=&v"(f) : "v"(df) : "memory");
        const bool ok = none || (f.x >= call && f.y >= call && f.z >= call && f.w >= call);
        if (__all(ok)) break;
        if (give_up(tmo, call, t0)) {
          if (lane == 0) p.out[rank][(rank & 3) * 4096 + ((rank >> 2) ? 31 * 128 : 0)] = __builtin_nanf("");
          break;
        }
        __builtin_amdgcn_s_sleep(2);
      }
      STAMP(5);
    }
  }
}

}

extern "C" void kernel_launch(int rank, int nranks, ncclComm_t comm,
                              hipStream_t stream,
                              void* const* d_in, const int* in_sizes, int n_in,
                              void* d_out, int out_size,
                              const int* in_dtypes, int out_dtype,
                              void* const* d_out_all,
                              void* const* d_ws_all, size_t ws_bytes) {
  (void)comm; (void)in_sizes; (void)n_in; (void)d_out; (void)out_size; (void)in_dtypes; (void)out_dtype;
  if (nranks != R || (unsigned)rank >= (unsigned)R || ws_bytes < WS_USED) {
    fprintf(stderr, "unsupported configuration\n");
    return;
  }
  Params p;
  p.x = (const int8_t*)d_in[0];
  p.w = (const int8_t*)d_in[1];
  p.sx = (const float*)d_in[2];
  p.sw = (const float*)d_in[3];
  for (int i = 0; i < R; ++i) { p.out[i] = (float*)d_out_all[i]; p.ws[i] = (char*)d_ws_all[i]; }
  p.rank = rank;
  fused_kernel<<<NWG, NTHR, 0, stream>>>(p);
  HIP_CHECK(hipGetLastError());
}
```

```cpp
#include <hip/hip_runtime.h>
#include <rccl/rccl.h>
#include <cstdio>
#include <cstdint>

#define HIP_CHECK(expr) do { hipError_t _e = (expr); if (_e != hipSuccess) { \
    fprintf(stderr, "HIP error %s at %s:%d\n", hipGetErrorString(_e), __FILE__, __LINE__); \
    return; } } while (0)

#define PPX 2
#define PULL_WIN 3
#define RING_D 3
#define OUT_AUX 17
#define ST_SLEEP 2
#define PUSH0 1
#define PUSH_WIN 1
#define POLL_SLEEP 4
#define NF 7
#define LG1 4

namespace {

typedef int      i32x4  __attribute__((ext_vector_type(4)));
typedef int      i32x16 __attribute__((ext_vector_type(16)));
typedef unsigned u32x4  __attribute__((ext_vector_type(4)));

constexpr int R = 8;
constexpr int GN = 16384;
constexpr int KS = 512;
constexpr int NWG = 256, NTHR = 512;

constexpr unsigned XS_BYTES = 8u << 20;
constexpr unsigned WS_BYTES = 16u << 20;
constexpr unsigned XE = XS_BYTES + WS_BYTES;
constexpr unsigned WE = XE + (2u << 20);
constexpr unsigned OS = WE + (8u << 20);
constexpr unsigned BUF_BYTES = OS + (24u << 20);
constexpr unsigned CTL = 2 * BUF_BYTES;
constexpr unsigned XFLAG = CTL;
constexpr unsigned WFLAG = XFLAG + 8 * 128 * 4;
constexpr unsigned DONE = WFLAG + 32 * 64 * 4;
constexpr unsigned SFLAG = DONE + 1024;
constexpr unsigned GFLAG = SFLAG + 16384;
constexpr unsigned CNT = GFLAG + 2048;
constexpr unsigned TMO = CNT + 1024;
constexpr unsigned QUE = TMO + 1024;
constexpr unsigned DBG = QUE + 1024;
constexpr unsigned WS_USED = DBG + 2 * 256 * 16 * 8;

constexpr int NB = 16 - NF, T_0 = PUSH0 ? 24 : 0, T_A = 160, T_B = 224, T_X = T_B + 8 * NB, T_O = T_X + 32, N_OTHER = T_O + 512, N_PART = T_O + 56 * NF;
constexpr int STAGE = 24576, NSLOT = 6, NSTAGE = 64;
constexpr unsigned long long DEADLINE = 50000000ull;
constexpr int AUX_SYS = 17;

struct Params {
  const int8_t* x; const int8_t* w; const float* sx; const float* sw;
  float* out; char* ws[R];
  int rank;
};

#define WAIT_VM(n) asm volatile("s_waitcnt vmcnt(" #n ")" ::: "memory")
#define WAIT_VM_(n) WAIT_VM(n)
#define RAW_BARRIER() do { asm volatile("s_waitcnt lgkmcnt(0)" ::: "memory"); __builtin_amdgcn_s_barrier(); asm volatile("" ::: "memory"); } while (0)
#define LDS3(p) ((__attribute__((address_space(3))) void*)(p))

#define STAMP(i) do {} while (0)
#define STAMPW(i) do {} while (0)

__device__ __forceinline__ __amdgpu_buffer_rsrc_t make_rsrc(const void* p, unsigned bytes) {
  return __builtin_amdgcn_make_buffer_rsrc((void*)p, 0, bytes, 0x00020000);
}
__device__ __forceinline__ void st_sys(void* p, unsigned v) {
  __hip_atomic_store((unsigned*)p, v, __ATOMIC_RELAXED, __HIP_MEMORY_SCOPE_SYSTEM);
}
__device__ __forceinline__ bool timed_out(unsigned* tmo, unsigned call) {
  return __builtin_amdgcn_readfirstlane(__hip_atomic_load(tmo, __ATOMIC_RELAXED, __HIP_MEMORY_SCOPE_AGENT)) == call;
}
__device__ __forceinline__ bool give_up(unsigned* tmo, unsigned call, unsigned long long t0) {
  if (timed_out(tmo, call) || __builtin_amdgcn_s_memrealtime() - t0 > DEADLINE) {
    __hip_atomic_store(tmo, call, __ATOMIC_RELAXED, __HIP_MEMORY_SCOPE_AGENT);
    return true;
  }
  return false;
}
__device__ __forceinline__ int xmask(int j) { return j + 1 + (j >= 3); }
__device__ __forceinline__ unsigned min4(u32x4 f) { return min(min(f.x, f.y), min(f.z, f.w)); }

__device__ __forceinline__ void transpose4(unsigned d0, unsigned d1, unsigned d2, unsigned d3,
                                           unsigned& o0, unsigned& o1, unsigned& o2, unsigned& o3) {
  unsigned t0 = __builtin_amdgcn_perm(d1, d0, 0x05010400u);
  unsigned t1 = __builtin_amdgcn_perm(d1, d0, 0x07030602u);
  unsigned t2 = __builtin_amdgcn_perm(d3, d2, 0x05010400u);
  unsigned t3 = __builtin_amdgcn_perm(d3, d2, 0x07030602u);
  o0 = __builtin_amdgcn_perm(t2, t0, 0x05040100u);
  o1 = __builtin_amdgcn_perm(t2, t0, 0x07060302u);
  o2 = __builtin_amdgcn_perm(t3, t1, 0x05040100u);
  o3 = __builtin_amdgcn_perm(t3, t1, 0x07060302u);
}

__device__ __forceinline__ float silu_dq(int acc, float s) {
  float y = (float)acc * s;
  float c = __builtin_amdgcn_fmed3f(y, -60.0f, 60.0f);
  float e = __builtin_amdgcn_exp2f(c * -1.44269504088896341f);
  return y * __builtin_amdgcn_rcpf(1.0f + e);
}

struct Task { int type, a, b, src; };

__global__ void __launch_bounds__(NTHR, 2) fused_kernel(Params p) {
  __shared__ __attribute__((aligned(16))) char lds[STAGE * NSLOT];
  const int lane = threadIdx.x & 63;
  const int wave = __builtin_amdgcn_readfirstlane(threadIdx.x >> 6);
  const int bid = blockIdx.x;
  const int rank = p.rank, hf = rank >> 2;
  char* const myws = p.ws[rank];
  unsigned* const tmo = (unsigned*)(myws + TMO);
  unsigned* const cnt = (unsigned*)(myws + CNT) + bid;
  const unsigned cnt_v = __hip_atomic_load(cnt, __ATOMIC_RELAXED, __HIP_MEMORY_SCOPE_AGENT);
  const int h = lane >> 5, c31 = lane & 31;
  const int xcd = bid & 7, wi = bid >> 3;
  const __amdgpu_buffer_rsrc_t mrs = make_rsrc(myws, WS_USED);
  const __amdgpu_buffer_rsrc_t ors = make_rsrc(p.out, 512u * GN * 4u);

  char* const lws = p.ws[lane & 7];
  const int pu = (bid - 64) >> 3, pd = (bid + pu) & 7, prq = pu / 6, plrb = 2 + pu - 6 * prq;
  const bool active = wave < 5 || (PUSH0 && wave == 5 && bid >= 64 && pd != rank);
  if (active) {
    const int r8 = lane >> 3, k8 = lane & 7;
    const int u = (wave - 1) * 256 + bid;
    const bool isx = wave == 0 || wave == 5;
    const char* src = wave == 0 ? (const char*)p.x + (size_t)((bid >> 4) * 256 + (bid & 3) * 64) * KS + ((bid >> 2) & 3) * 128
                    : wave == 5 ? (const char*)p.x + (size_t)(512 * (pd ^ xmask(plrb - 2)) + 256 * (pd >> 2) + 64 * prq) * KS
                                : (const char*)p.w + (size_t)(64 * (u & 7)) * GN + (u >> 3) * 128;
    const unsigned stride = isx ? KS : GN;
    const __amdgpu_buffer_rsrc_t srs = make_rsrc(src, 63 * stride + 128);
    const unsigned voff = r8 * stride + ((isx ? k8 ^ r8 : k8) * 16);
#pragma unroll
    for (int j = 0; j < 8; ++j)
      __builtin_amdgcn_raw_ptr_buffer_load_lds(srs, LDS3(lds + wave * 8192 + j * 1024), 16, voff, j * 8 * stride, 0, 0);
  }
  asm volatile("" ::: "memory");
  const unsigned call = __builtin_amdgcn_readfirstlane(cnt_v) + 1u;
  const unsigned par = (call & 1u) * BUF_BYTES;
  if (active) {
    WAIT_VM(0);
    const char* l = lds + wave * 8192;
    unsigned flag = 0;
    if (wave == 0) {
      const int rb = bid >> 4, b = bid & 15;
      auto lrb_of = [&](int d) { const int m = (rb >> 1) ^ d; return m == 0 ? (rb & 1) : (((m & 3) && (rb & 1) == (d >> 2)) ? 1 + m - (m > 4) : -1); };
      const int lrb = lrb_of(rank), lrbl = lrb_of(lane & 7);
      const unsigned goff = lrb >= 0 ? par + (unsigned)((lrb * 8 + rank) * 16 + (b >> 2) * 4) * 8192 + (b & 3) * 2048 + lane * 16 : 0xf0000000u;
      if (lrbl >= 0) flag = lane == rank ? XFLAG + ((lrbl * 8 + rank) * 16 + b) * 4 : SFLAG + rank * 2048 + 1024 + (lrbl * 16 + b) * 4;
#pragma unroll
      for (int j = 0; j < 8; ++j) {
        const u32x4 v = *(const u32x4*)(l + c31 * 128 + (((2 * (j >> 1) + h) ^ (c31 & 7)) * 16) + (j & 1) * 4096);
        __builtin_amdgcn_raw_buffer_store_b128(v, mrs, par + XE + bid * 8192 + j * 1024 + lane * 16, 0, AUX_SYS);
        __builtin_amdgcn_raw_buffer_store_b128(v, mrs, goff + (j >> 1) * 8192 + (j & 1) * 1024, 0, AUX_SYS);
      }
    } else if (wave == 5) {
      const __amdgpu_buffer_rsrc_t drs = make_rsrc(p.ws[pd], CTL);
      const unsigned goff = par + (unsigned)((plrb * 8 + rank) * 16) * 8192 + prq * 2048 + lane * 16;
      if (lane == pd) flag = XFLAG + ((plrb * 8 + rank) * 16 + prq) * 4;
#pragma unroll
      for (int j = 0; j < 8; ++j) {
        const u32x4 v = *(const u32x4*)(l + c31 * 128 + (((2 * (j >> 1) + h) ^ (c31 & 7)) * 16) + (j & 1) * 4096);
        __builtin_amdgcn_raw_buffer_store_b128(v, drs, goff + (j >> 1) * 8192 + (j & 1) * 1024, 0, AUX_SYS);
        WAIT_VM_(PUSH_WIN);
      }
    } else {
      const int u = (wave - 1) * 256 + bid, cb = (u >> 3) & 31, kp = u & 7;
      const bool mine = (wave - 1) == (rank & 3);
      const unsigned goff = mine ? par + XS_BYTES + (unsigned)((cb * 8 + rank) * 16 + 2 * kp) * 4096 + lane * 16 : 0xf0000000u;
      if ((lane & 3) == wave - 1) flag = lane == rank ? WFLAG + ((cb * 8 + rank) * 8 + kp) * 4 : SFLAG + rank * 2048 + bid * 4;
#pragma unroll
      for (int kk = 0; kk < 2; ++kk) {
        unsigned o[4][4];
#pragma unroll
        for (int q = 0; q < 4; ++q) {
          const unsigned* lp = (const unsigned*)(l + (kk * 32 + h * 16 + q * 4) * 128 + c31 * 4);
          transpose4(lp[0], lp[32], lp[64], lp[96], o[0][q], o[1][q], o[2][q], o[3][q]);
        }
#pragma unroll
        for (int b = 0; b < 4; ++b) {
          const u32x4 v = {o[b][0], o[b][1], o[b][2], o[b][3]};
          __builtin_amdgcn_raw_buffer_store_b128(v, mrs, par + WE + u * 8192 + (kk * 4 + b) * 1024 + lane * 16, 0, AUX_SYS);
          __builtin_amdgcn_raw_buffer_store_b128(v, mrs, goff + (kk * 4 + b) * 1024, 0, AUX_SYS);
        }
      }
    }
    WAIT_VM(0);
    if (flag && lane < 8) st_sys(lws + flag, call);
  }
  const unsigned ctl = (unsigned)(size_t)(__attribute__((address_space(3))) char*)(lds + 131072 + (wave & 3) * 16);
  auto ctl_st = [&](int slot, int v) { asm volatile("ds_write_b32 %0, %1" :: "v"(ctl + slot * 4), "v"(v) : "memory"); };
  auto ctl_ld = [&](int slot) {
    int v; asm volatile("ds_read_b32 %0, %1\n\ts_waitcnt lgkmcnt(0)" : "=&v"(v) : "v"(ctl + slot * 4) : "memory");
    return __builtin_amdgcn_readfirstlane(v);
  };
  if (wave >= 4 && lane < 4) ctl_st(lane, 0);
  RAW_BARRIER();
  if (wave == 7 && lane == 0) __hip_atomic_store(cnt, call, __ATOMIC_RELAXED, __HIP_MEMORY_SCOPE_AGENT);

  if (wi < 7) {
    const int pair = wave & 3;
    if (pair >= PPX) return;
    const int peer = wi == 0 ? rank ^ 4 : rank ^ xmask(wi - 1);
    const int ntask = wi == 0 ? N_PART : N_OTHER;
    constexpr int cn = 8 * PPX;
    const int cw = T_0 + pair * 8 + xcd;
    char* const pws = p.ws[peer];
    char* const ring = lds + pair * 32768;
    auto decode = [&](int i) {
      Task t; t.src = peer;
      int cp = 0;
      if (i < T_A) {
        const int kq = i / 40, u = i - kq * 40;
        if (u < 24) { const int rq = u / 6; t.type = 1; t.a = 2 + u - 6 * rq; t.b = kq * 4 + rq; return t; }
        t.type = 0; cp = (u - 24) & 7; t.b = 2 * kq + ((u - 24) >> 3);
      } else if (i < T_B) { t.type = 0; cp = 8 + (i & 7); t.b = (i - T_A) >> 3;
      } else if (i < T_X) { t.type = 0; t.b = (i - T_B) / NB; cp = 16 + i - T_B - NB * t.b; }
      else if (wi != 0 ? i < T_O : i >= N_PART - 32) {
        const int u = i - (wi != 0 ? T_X : N_PART - 32);
        t.type = 1; t.a = u & 1; t.b = u >> 1; return t;
      } else if (wi != 0) {
        t.type = 3; t.b = (i - T_O) & 15; cp = (i - T_O) >> 4; t.a = (peer >> 2) ? 31 - cp : cp; return t;
      } else {
        int j = i - T_X, js = 0, gs = LG1;
        if (j >= LG1 * 56) { j -= LG1 * 56; js = LG1; gs = NF - LG1; }
        const int kp = j / (gs * 7), u = j - kp * gs * 7, jb = u / 7, k = u - jb * 7;
        cp = 32 - NF + js + jb; t.b = kp;
        if (k == 0) t.type = 0; else { t.type = 2; t.src = rank ^ xmask(k - 1); }
      }
      t.a = hf ? 31 - cp : cp;
      return t;
    };
    if (wave >= 4) {
      const __amdgpu_buffer_rsrc_t prs = make_rsrc(pws, CTL);
      int pend = -1;
#pragma unroll 1
      for (int k = 0;; ++k) {
        const int slot = k & 3;
        int c = ctl_ld(slot);
        if (c <= 0 && c != -2) {
          if (pend >= 0) { WAIT_VM(0); ctl_st(pend, -1); pend = -1; }
          while ((c = ctl_ld(slot)) <= 0 && c != -2) __builtin_amdgcn_s_sleep(1);
        }
        if (c == -2) break;
        const Task t = decode(c - 1);
        unsigned soff;
        if (t.type == 0) soff = WE + (unsigned)(((rank & 3) * 32 + t.a) * 8 + t.b) * 8192;
        else if (t.type == 1) soff = XE + (unsigned)((t.a < 2 ? 2 * rank + t.a : 2 * (rank ^ xmask(t.a - 2)) + hf) * 16 + t.b) * 8192;
        else if (t.type == 2) soff = XS_BYTES + (unsigned)((t.a * 8 + t.src) * 16 + 2 * t.b) * 4096;
        else soff = OS + (unsigned)(((wi - 1) * 32 + t.a) * 16 + t.b) * 8192;
        soff += par;
#pragma unroll
        for (int j = 0; j < 8; ++j) {
          __builtin_amdgcn_raw_ptr_buffer_load_lds(prs, LDS3(ring + slot * 8192 + j * 1024), 16, lane * 16, soff + j * 1024, 0, AUX_SYS);
          WAIT_VM_(PULL_WIN);
          if (j == (PULL_WIN > 0 ? PULL_WIN - 1 : 0) && pend >= 0) ctl_st(pend, -1);
        }
        pend = slot;
      }
      if (pend >= 0) { WAIT_VM(0); ctl_st(pend, -1); }
    } else {
      const unsigned qoff = lane == 0 ? QUE + peer * 128 : 0xfffffff0u;
      const unsigned tbase = (call - 1u) * (unsigned)(ntask - T_0 - (RING_D - 1) * cn) - (unsigned)(T_0 + RING_D * cn);
      bool staged = false, given = false, ended = false, have = true, dead = false;
      unsigned tiles = 0;
      unsigned nxt = cw;
      unsigned long long held = 0;
      int ka = 0, kf = 0;
      unsigned long long t0 = __builtin_amdgcn_s_memrealtime();
      auto ready = [&](unsigned i) {
        if (dead) return true;
        if (!staged) {
          u32x4 f0, f1;
          asm volatile("global_load_dwordx4 %0, %2, off sc0 sc1\n\t"
                       "global_load_dwordx4 %1, %3, off sc0 sc1\n\t"
                       "s_waitcnt vmcnt(0)" : "=&v"(f0), "=&v"(f1)
                       : "v"(myws + SFLAG + peer * 2048 + lane * 16), "v"(myws + SFLAG + peer * 2048 + 1024 + c31 * 16) : "memory");
          if (!(staged = __all(min(min4(f0), min4(f1)) >= call))) return false;
        }
        if (wi == 0) {
          if (i < T_X || given) return true;
          const bool skip = ((lane & 3) == (rank & 3)) || (hf ? (lane >> 3) >= NF : (lane >> 3) < 8 - NF);
          u32x4 f0, f1;
          asm volatile("global_load_dwordx4 %0, %2, off sc0 sc1\n\t"
                       "global_load_dwordx4 %1, %2, off offset:16 sc0 sc1\n\t"
                       "s_waitcnt vmcnt(0)" : "=&v"(f0), "=&v"(f1) : "v"(myws + GFLAG + lane * 32) : "memory");
          return given = __all(skip || min(min4(f0), min4(f1)) >= call);
        }
        if (i < T_O) return true;
        const unsigned cp = (i - T_O) >> 4, bit = 1u << ((peer >> 2) ? 31 - cp : cp);
        if (tiles & bit) return true;
        unsigned f;
        asm volatile("global_load_dword %0, %1, off sc0 sc1\n\ts_waitcnt vmcnt(0)" : "=&v"(f) : "v"(myws + DONE + peer * 128 + c31 * 4) : "memory");
        tiles = (unsigned)__ballot(f >= call);
        return (tiles & bit) != 0;
      };
      auto assign = [&]() {
        bool any = false;
        while (!ended && ka < kf + RING_D) {
          if (!have) { nxt = (unsigned)__builtin_amdgcn_readfirstlane(__builtin_amdgcn_raw_ptr_buffer_atomic_add_i32(1, mrs, qoff, 0, 0)) - tbase; have = true; }
          if (nxt >= (unsigned)ntask) { ctl_st(ka & 3, -2); ended = true; break; }
          if (!ready(nxt)) break;
          ctl_st(ka & 3, nxt + 1);
          held = (held & ~(0xffffull << (16 * (ka & 3)))) | ((unsigned long long)nxt << (16 * (ka & 3)));
          ++ka; any = true;
          if (ka < RING_D) nxt += cn; else have = false;
        }
        return any;
      };
#pragma unroll 1
      for (unsigned polls = 1;; ++polls) {
        bool progress = assign();
        if (kf < ka && ctl_ld(kf & 3) == -1) {
          const bool draw = !have && !ended;
          const int tk = __builtin_amdgcn_raw_ptr_buffer_atomic_add_i32(1, mrs, draw ? qoff : 0xfffffff0u, 0, 0);
          unsigned tkv;
          const Task t = decode((int)((held >> (16 * (kf & 3))) & 0xffff));
          const char* l = ring + (kf & 3) * 8192 + lane * 16;
          u32x4 v[8];
#pragma unroll
          for (int j = 0; j < 8; ++j) v[j] = *(const u32x4*)(l + j * 1024);
          unsigned flag = 0;
          if (t.type == 3) {
            const unsigned off = (unsigned)((peer >> 2) * 256 + t.b * 16 + 4 * h) * (GN * 4) + ((peer & 3) * 4096 + t.a * 128) * 4 + c31 * 16;
            const unsigned poison = dead ? 0x7fc00000u : 0u;
#pragma unroll
            for (int j = 0; j < 8; ++j)
              __builtin_amdgcn_raw_buffer_store_b128(v[j] | poison, ors, off + ((j & 3) + 8 * (j >> 2)) * (GN * 4), 0, OUT_AUX);
            __builtin_amdgcn_sched_barrier(0);
            tkv = __builtin_amdgcn_readfirstlane(tk);
          } else {
            unsigned off, s1;
            if (t.type == 1) {
              off = (unsigned)((t.a * 8 + peer) * 16 + (t.b >> 2) * 4) * 8192 + (t.b & 3) * 2048; s1 = 8192;
              flag = XFLAG + ((t.a * 8 + peer) * 16 + t.b) * 4;
            } else {
              off = XS_BYTES + (unsigned)((t.a * 8 + t.src) * 16 + 2 * t.b) * 4096; s1 = 2048;
              flag = WFLAG + ((t.a * 8 + t.src) * 8 + t.b) * 4;
            }
            off += par + lane * 16;
#pragma unroll
            for (int j = 0; j < 8; ++j)
              __builtin_amdgcn_raw_buffer_store_b128(v[j], mrs, off + (j >> 1) * s1 + (j & 1) * 1024, 0, AUX_SYS);
            __builtin_amdgcn_sched_barrier(0);
            tkv = __builtin_amdgcn_readfirstlane(tk);
          }
          ++kf;
          if (draw) { nxt = tkv - tbase; have = true; }
          assign();
          if (flag) {
            WAIT_VM(0);
            if (lane == 0) st_sys(myws + flag, call);
            if (lane == 1 && wi != 0 && t.type == 0 && (hf ? t.a >= 32 - NF : t.a < NF))
              st_sys(p.ws[rank ^ 4] + GFLAG + (((t.a & 7) * 8 + peer) * 8 + t.b) * 4, call);
          }
          progress = true;
        }
        if (ended && kf == ka) break;
        if (progress) { polls = 0; t0 = __builtin_amdgcn_s_memrealtime(); }
        else {
          if ((polls & 63) == 0 && give_up(tmo, call, t0)) dead = true;
          __builtin_amdgcn_s_sleep(ST_SLEEP);
        }
      }
      STAMPW(8 + wave);
    }
    return;
  }

  const int njob = (wi >= 8 && wi < 15) ? 2 : 1;
#pragma unroll 1
  for (int job = 0; job < njob; ++job) {
    int lrb, jc;
    if (wi == 7 || job) { lrb = wi & 1; jc = (wi >> 1) & 3; }
    else { jc = (wi - 8) / 6; lrb = 2 + (wi - 8) - 6 * jc; }
    const int cp = xcd + 8 * jc;
    const int cb = hf ? 31 - cp : cp;
    auto stage_id = [&](int q) {
      if (q < 8) return rank * 8 + q;
      q -= 8;
      if (lrb >= 2) { const int kp = q / 7, k = q - 7 * kp; return (rank ^ (k == 0 ? 4 : xmask(k - 1))) * 8 + kp; }
      if (q >= 48) return (rank ^ 4) * 8 + q - 48;
      const int kp = q / 6;
      return (rank ^ xmask(q - 6 * kp)) * 8 + kp;
    };
    const int my_id = stage_id(lane);
    const unsigned* const xf = (const unsigned*)(myws + XFLAG) + lrb * 128 + (my_id >> 1) * 4;
    const unsigned* const wf = (const unsigned*)(myws + WFLAG) + cb * 64 + my_id;
    int ready = 0;
    auto ensure = [&](int q) {
      if (ready > q) return;
      const unsigned long long t0 = __builtin_amdgcn_s_memrealtime();
      unsigned polls = 0;
      for (;;) {
        u32x4 fx; unsigned fw;
        asm volatile("global_load_dwordx4 %0, %2, off sc0 sc1\n\t"
                     "global_load_dword %1, %3, off sc0 sc1\n\t"
                     "s_waitcnt vmcnt(0)"
                     : "=&v"(fx), "=&v"(fw) : "v"(xf), "v"(wf) : "memory");
        const unsigned long long m = __ballot(min(min4(fx), fw) >= call);
        ready = (~m == 0ull) ? 64 : __builtin_ctzll(~m);
        if (ready > q) return;
        if ((++polls & 15) == 0 && give_up(tmo, call, t0)) { ready = 64; return; }
        __builtin_amdgcn_s_sleep(POLL_SLEEP);
      }
    };
    const unsigned a_base = par + (unsigned)lrb * (1u << 20) + wave * 2048;
    const unsigned b_base = par + XS_BYTES + (unsigned)cb * (512u << 10) + wave * 1024;
    auto issue = [&](int q, int slot) {
      const int id = stage_id(q);
      char* l = lds + slot * STAGE;
      auto* l0 = LDS3(l + wave * 2048);
      auto* l1 = LDS3(l + wave * 2048 + 1024);
      auto* l2 = LDS3(l + 16384 + wave * 1024);
      __builtin_amdgcn_raw_ptr_buffer_load_lds(mrs, l0, 16, lane * 16, a_base + id * 16384, 0, AUX_SYS);
      __builtin_amdgcn_raw_ptr_buffer_load_lds(mrs, l1, 16, lane * 16, a_base + id * 16384 + 1024, 0, AUX_SYS);
      __builtin_amdgcn_raw_ptr_buffer_load_lds(mrs, l2, 16, lane * 16, b_base + id * 8192, 0, AUX_SYS);
    };
    i32x16 acc[4];
#pragma unroll
    for (int b = 0; b < 4; ++b)
#pragma unroll
      for (int i = 0; i < 16; ++i) acc[b][i] = 0;
    i32x4 fa[2][2], fb[2][2][4];
    auto rd = [&](int buf, int slot) {
      const char* l = lds + slot * STAGE + lane * 16;
#pragma unroll
      for (int ks = 0; ks < 2; ++ks) {
        fa[buf][ks] = *(const i32x4*)(l + ks * 8192 + wave * 1024);
#pragma unroll
        for (int b = 0; b < 4; ++b) fb[buf][ks][b] = *(const i32x4*)(l + 16384 + ks * 4096 + b * 1024);
      }
    };
    auto mm = [&](int buf) {
#pragma unroll
      for (int ks = 0; ks < 2; ++ks)
#pragma unroll
        for (int b = 0; b < 4; ++b) acc[b] = __builtin_amdgcn_mfma_i32_32x32x32_i8(fa[buf][ks], fb[buf][ks][b], acc[b], 0, 0, 0);
    };
    auto nx = [&](int s) { return s == NSLOT - 1 ? 0 : s + 1; };

    if (wave == 0) ensure(NSLOT - 1);
    RAW_BARRIER();
    STAMP(1);
#pragma unroll
    for (int q = 0; q < NSLOT; ++q) issue(q, q);
    WAIT_VM(15);
    RAW_BARRIER();
    rd(0, 0);
    int s0 = 0;
#pragma unroll 1
    for (int i = 0; i < NSTAGE - NSLOT; i += 2) {
      if (wave == 0) ensure(i + NSLOT);
      WAIT_VM(12); RAW_BARRIER();
      issue(i + NSLOT, s0); rd(1, nx(s0)); mm(0);
      s0 = nx(s0);
      if (wave == 0) ensure(i + NSLOT + 1);
      WAIT_VM(12); RAW_BARRIER();
      issue(i + NSLOT + 1, s0); rd(0, nx(s0)); mm(1);
      s0 = nx(s0);
    }
    STAMP(2);
    WAIT_VM(12); RAW_BARRIER(); rd(1, nx(s0)); mm(0); s0 = nx(s0);
    WAIT_VM(9);  RAW_BARRIER(); rd(0, nx(s0)); mm(1); s0 = nx(s0);
    WAIT_VM(6);  RAW_BARRIER(); rd(1, nx(s0)); mm(0); s0 = nx(s0);
    WAIT_VM(3);  RAW_BARRIER(); rd(0, nx(s0)); mm(1); s0 = nx(s0);
    WAIT_VM(0);  RAW_BARRIER(); rd(1, nx(s0)); mm(0);
    mm(1);
    STAMP(3);

    {
      const float scale = p.sx[0] * p.sw[0];
      const unsigned poison = timed_out(tmo, call) ? 0x7fc00000u : 0u;
      const unsigned ooff = (unsigned)(lrb * 256 + wave * 32 + 4 * h) * (GN * 4) + ((rank & 3) * 4096 + cb * 128) * 4 + c31 * 16;
      const unsigned soff = par + OS + (unsigned)(((lrb - 2) * 32 + cb) * 8 + wave) * 16384 + lane * 16;
#pragma unroll
      for (int i = 0; i < 16; ++i) {
        u32x4 o;
#pragma unroll
        for (int b = 0; b < 4; ++b) o[b] = __builtin_bit_cast(unsigned, silu_dq(acc[b][i], scale)) | poison;
        if (lrb < 2) __builtin_amdgcn_raw_buffer_store_b128(o, ors, ooff + ((i & 3) + 8 * (i >> 2)) * (GN * 4), 0, OUT_AUX);
        else __builtin_amdgcn_raw_buffer_store_b128(o, mrs, soff + i * 1024, 0, AUX_SYS);
      }
    }
    WAIT_VM(0);
    RAW_BARRIER();
    STAMP(4);
    if (wave == 0 && lane == 0 && lrb >= 2) st_sys(p.ws[rank ^ xmask(lrb - 2)] + DONE + (rank * 32 + cb) * 4, call);
  }
}

}

extern "C" void kernel_launch(int rank, int nranks, ncclComm_t comm,
                              hipStream_t stream,
                              void* const* d_in, const int* in_sizes, int n_in,
                              void* d_out, int out_size,
                              const int* in_dtypes, int out_dtype,
                              void* const* d_out_all,
                              void* const* d_ws_all, size_t ws_bytes) {
  (void)comm; (void)in_sizes; (void)n_in; (void)d_out_all; (void)out_size; (void)in_dtypes; (void)out_dtype;
  if (nranks != R || (unsigned)rank >= (unsigned)R || ws_bytes < WS_USED) {
    fprintf(stderr, "unsupported configuration\n");
    return;
  }
  Params p;
  p.x = (const int8_t*)d_in[0];
  p.w = (const int8_t*)d_in[1];
  p.sx = (const float*)d_in[2];
  p.sw = (const float*)d_in[3];
  p.out = (float*)d_out;
  for (int i = 0; i < R; ++i) p.ws[i] = (char*)d_ws_all[i];
  p.rank = rank;
  fused_kernel<<<NWG, NTHR, 0, stream>>>(p);
  HIP_CHECK(hipGetLastError());
}
```

```cpp
#include <hip/hip_runtime.h>
#include <rccl/rccl.h>
#include <cstdio>
#include <cstdint>

#define HIP_CHECK(expr) do { hipError_t _e = (expr); if (_e != hipSuccess) { \
    fprintf(stderr, "HIP error %s at %s:%d\n", hipGetErrorString(_e), __FILE__, __LINE__); \
    return; } } while (0)

#define PPX 2
#define PULL_WIN 2
#define RING_D 3
#define OUT_AUX 17
#define ST_SLEEP 2
#define STG_DELAY 10
#define PUSH_WIN 2
#define TAILN 8
#define POLL_SLEEP 4
#define NF 7
#define LG1 4

namespace {

typedef int      i32x4  __attribute__((ext_vector_type(4)));
typedef int      i32x16 __attribute__((ext_vector_type(16)));
typedef unsigned u32x4  __attribute__((ext_vector_type(4)));

constexpr int R = 8;
constexpr int GN = 16384;
constexpr int KS = 512;
constexpr int NWG = 256, NTHR = 512;

constexpr unsigned XS_BYTES = 8u << 20;
constexpr unsigned WS_BYTES = 16u << 20;
constexpr unsigned XE = XS_BYTES + WS_BYTES;
constexpr unsigned WE = XE + (2u << 20);
constexpr unsigned OS = WE + (8u << 20);
constexpr unsigned BUF_BYTES = OS + (24u << 20);
constexpr unsigned CTL = 2 * BUF_BYTES;
constexpr unsigned XFLAG = CTL;
constexpr unsigned WFLAG = XFLAG + 8 * 128 * 4;
constexpr unsigned DONE = WFLAG + 32 * 64 * 4;
constexpr unsigned SFLAG = DONE + 1024;
constexpr unsigned GFLAG = SFLAG + 16384;
constexpr unsigned CNT = GFLAG + 2048;
constexpr unsigned TMO = CNT + 1024;
constexpr unsigned QUE = TMO + 1024;
constexpr unsigned DBG = QUE + 1024;
constexpr unsigned WS_USED = DBG + 2 * 256 * 16 * 8;

constexpr int NB = 16 - NF, T_0 = 24  , T_A = 160, T_B = 224, T_X = T_B + 8 * NB, T_O = T_X + 32, N_OTHER = T_O + 512, N_PART = T_O + 56 * NF;
constexpr int STAGE = 24576, NSLOT = 6, NSTAGE = 64;
constexpr unsigned long long DEADLINE = 50000000ull;
constexpr int AUX_SYS = 17;

struct Params {
  const int8_t* x; const int8_t* w; char* myws; int rank; int pad;
  const float* sx; const float* sw;
  float* out; char* ws[R];
};

#define WAIT_VM(n) asm volatile("s_waitcnt vmcnt(" #n ")" ::: "memory")
#define WAIT_VM_(n) WAIT_VM(n)
#define RAW_BARRIER() do { asm volatile("s_waitcnt lgkmcnt(0)" ::: "memory"); __builtin_amdgcn_s_barrier(); asm volatile("" ::: "memory"); } while (0)
#define LDS3(p) ((__attribute__((address_space(3))) void*)(p))

#define STAMP(i) do {} while (0)
#define STAMPW(i) do {} while (0)

__device__ __forceinline__ __amdgpu_buffer_rsrc_t make_rsrc(const void* p, unsigned bytes) {
  return __builtin_amdgcn_make_buffer_rsrc((void*)p, 0, bytes, 0x00020000);
}
__device__ __forceinline__ void st_sys(void* p, unsigned v) {
  __hip_atomic_store((unsigned*)p, v, __ATOMIC_RELAXED, __HIP_MEMORY_SCOPE_SYSTEM);
}
__device__ __forceinline__ bool timed_out(unsigned* tmo, unsigned call) {
  return __builtin_amdgcn_readfirstlane(__hip_atomic_load(tmo, __ATOMIC_RELAXED, __HIP_MEMORY_SCOPE_AGENT)) == call;
}
__device__ __forceinline__ bool give_up(unsigned* tmo, unsigned call, unsigned long long t0) {
  if (timed_out(tmo, call) || __builtin_amdgcn_s_memrealtime() - t0 > DEADLINE) {
    __hip_atomic_store(tmo, call, __ATOMIC_RELAXED, __HIP_MEMORY_SCOPE_AGENT);
    return true;
  }
  return false;
}
__device__ __forceinline__ int xmask(int j) { return j + 1 + (j >= 3); }
__device__ __forceinline__ unsigned min4(u32x4 f) { return min(min(f.x, f.y), min(f.z, f.w)); }

__device__ __forceinline__ void transpose4(unsigned d0, unsigned d1, unsigned d2, unsigned d3,
                                           unsigned& o0, unsigned& o1, unsigned& o2, unsigned& o3) {
  unsigned t0 = __builtin_amdgcn_perm(d1, d0, 0x05010400u);
  unsigned t1 = __builtin_amdgcn_perm(d1, d0, 0x07030602u);
  unsigned t2 = __builtin_amdgcn_perm(d3, d2, 0x05010400u);
  unsigned t3 = __builtin_amdgcn_perm(d3, d2, 0x07030602u);
  o0 = __builtin_amdgcn_perm(t2, t0, 0x05040100u);
  o1 = __builtin_amdgcn_perm(t2, t0, 0x07060302u);
  o2 = __builtin_amdgcn_perm(t3, t1, 0x05040100u);
  o3 = __builtin_amdgcn_perm(t3, t1, 0x07060302u);
}

__device__ __forceinline__ float silu_dq(int acc, float s) {
  float y = (float)acc * s;
  float c = __builtin_amdgcn_fmed3f(y, -60.0f, 60.0f);
  float e = __builtin_amdgcn_exp2f(c * -1.44269504088896341f);
  return y * __builtin_amdgcn_rcpf(1.0f + e);
}

__device__ __forceinline__ unsigned xs_off(int lrb, int src, int b) { return (unsigned)(lrb * 64 + (b >> 2) * 16 + src) * 16384 + (b & 3) * 2048; }
__device__ __forceinline__ unsigned xs_piece(int j) { return (j >> 2) * 131072 + ((j >> 1) & 1) * 8192 + (j & 1) * 1024; }
__device__ __forceinline__ unsigned ws_off(int cb, int src, int kp) { return XS_BYTES + (unsigned)(cb * 64 + kp * 8 + src) * 8192; }
__device__ __forceinline__ unsigned xe_off(int rb, int b) { return XE + (unsigned)((b * 2 + (rb & 1)) * 8 + (rb >> 1)) * 8192; }
__device__ __forceinline__ unsigned we_off(int q, int cb, int kp) { return WE + (unsigned)((q * 8 + kp) * 32 + cb) * 8192; }

struct Task { int type, a, b, src; };

__global__ void __launch_bounds__(NTHR, 2) fused_kernel(Params p) {
  __shared__ __attribute__((aligned(16))) char lds[STAGE * NSLOT];
  const int lane = threadIdx.x & 63;
  const int wave = __builtin_amdgcn_readfirstlane(threadIdx.x >> 6);
  const int bid = blockIdx.x;
  const int rank = p.rank, hf = rank >> 2;
  char* const myws = p.myws;
  const int8_t* const px = p.x;
  const int8_t* const pw = p.w;
  asm volatile("" :: "s"(px), "s"(pw), "s"(myws), "s"(rank));
  unsigned* const tmo = (unsigned*)(myws + TMO);
  unsigned* const cnt = (unsigned*)(myws + CNT) + bid;
  const unsigned cnt_v = __hip_atomic_load(cnt, __ATOMIC_RELAXED, __HIP_MEMORY_SCOPE_AGENT);
  const int h = lane >> 5, c31 = lane & 31;
  const int xcd = bid & 7, wi = bid >> 3;
  const __amdgpu_buffer_rsrc_t mrs = make_rsrc(myws, WS_USED);
  const __amdgpu_buffer_rsrc_t ors = make_rsrc(p.out, 512u * GN * 4u);

  char* const lws = p.ws[lane & 7];
  const int n = bid - 56, sc = (wave - 1) * 200 + n;
  const int pu = n >> 3, pd = (n + pu) & 7, prq = pu / 6, plrb = 2 + pu - 6 * prq;
  const bool active = n >= 0 && (wave == 0 ? (n < 192 && pd != rank) : sc < 1280);
  int kind = 0, ca = 0, cbk = 0, ckp = 0;
  if (wave > 0) {
    const int kq = sc / 192, u = sc < 768 ? sc - kq * 192 : sc - 768;
    if (sc < 768 && u >= 128) { kind = 1; ca = u & 15; cbk = kq * 4 + ((u - 128) >> 4); }
    else {
      kind = 2; ca = (u >> 4) & 3;
      if (sc < 768) { ckp = 2 * kq + (u >> 6); cbk = (u & 15) + (u & 8) * 2; } else { ckp = u >> 6; cbk = 8 + (u & 15); }
    }
  }
  if (active) {
    const int r8 = lane >> 3, k8 = lane & 7;
    const char* src = kind == 0 ? (const char*)px + (size_t)(512 * (pd ^ xmask(plrb - 2)) + 256 * (pd >> 2) + 64 * prq) * KS
                    : kind == 1 ? (const char*)px + (size_t)(ca * 256 + (cbk & 3) * 64) * KS + (cbk >> 2) * 128
                                : (const char*)pw + (size_t)(64 * ckp) * GN + ca * 4096 + cbk * 128;
    const unsigned stride = kind < 2 ? KS : GN;
    const __amdgpu_buffer_rsrc_t srs = make_rsrc(src, 63 * stride + 128);
    const unsigned voff = r8 * stride + ((kind < 2 ? k8 ^ r8 : k8) * 16);
    for (int d = 1; d < wave; ++d) __builtin_amdgcn_s_sleep(STG_DELAY);
#pragma unroll
    for (int j = 0; j < 8; ++j)
      __builtin_amdgcn_raw_ptr_buffer_load_lds(srs, LDS3(lds + wave * 8192 + j * 1024), 16, voff, j * 8 * stride, 0, 0);
  }
  asm volatile("" ::: "memory");
  const unsigned call = __builtin_amdgcn_readfirstlane(cnt_v) + 1u;
  const unsigned par = (call & 1u) * BUF_BYTES;
  if (active) {
    WAIT_VM(0);
    const char* l = lds + wave * 8192;
    unsigned flag = 0;
    if (kind == 1) {
      const int rb = ca, b = cbk;
      auto lrb_of = [&](int d) { const int m = (rb >> 1) ^ d; return m == 0 ? (rb & 1) : (((m & 3) && (rb & 1) == (d >> 2)) ? 1 + m - (m > 4) : -1); };
      const int lrb = lrb_of(rank), lrbl = lrb_of(lane & 7);
      const unsigned goff = lrb >= 0 ? par + xs_off(lrb, rank, b) + lane * 16 : 0xf0000000u;
      if (lrbl >= 0) flag = lane == rank ? XFLAG + ((lrbl * 8 + rank) * 16 + b) * 4 : SFLAG + rank * 2048 + 1024 + (lrbl * 16 + b) * 4;
#pragma unroll
      for (int j = 0; j < 8; ++j) {
        const u32x4 v = *(const u32x4*)(l + c31 * 128 + (((2 * (j >> 1) + h) ^ (c31 & 7)) * 16) + (j & 1) * 4096);
        __builtin_amdgcn_raw_buffer_store_b128(v, mrs, par + xe_off(rb, b) + j * 1024 + lane * 16, 0, AUX_SYS);
        __builtin_amdgcn_raw_buffer_store_b128(v, mrs, goff + xs_piece(j), 0, AUX_SYS);
      }
    } else if (kind == 0) {
      const __amdgpu_buffer_rsrc_t drs = make_rsrc(p.ws[pd], CTL);
      const unsigned goff = par + xs_off(plrb, rank, prq) + lane * 16;
      if (lane == pd) flag = XFLAG + ((plrb * 8 + rank) * 16 + prq) * 4;
#pragma unroll
      for (int j = 0; j < 8; ++j) {
        const u32x4 v = *(const u32x4*)(l + c31 * 128 + (((2 * (j >> 1) + h) ^ (c31 & 7)) * 16) + (j & 1) * 4096);
        __builtin_amdgcn_raw_buffer_store_b128(v, drs, goff + xs_piece(j), 0, AUX_SYS);
        WAIT_VM_(PUSH_WIN);
      }
    } else {
      const unsigned goff = ca == (rank & 3) ? par + ws_off(cbk, rank, ckp) + lane * 16 : 0xf0000000u;
      if ((lane & 3) == ca) flag = lane == rank ? WFLAG + ((cbk * 8 + rank) * 8 + ckp) * 4 : SFLAG + rank * 2048 + (cbk * 8 + ckp) * 4;
#pragma unroll
      for (int kk = 0; kk < 2; ++kk) {
        unsigned o[4][4];
#pragma unroll
        for (int q = 0; q < 4; ++q) {
          const unsigned* lp = (const unsigned*)(l + (kk * 32 + h * 16 + q * 4) * 128 + c31 * 4);
          transpose4(lp[0], lp[32], lp[64], lp[96], o[0][q], o[1][q], o[2][q], o[3][q]);
        }
#pragma unroll
        for (int b = 0; b < 4; ++b) {
          const u32x4 v = {o[b][0], o[b][1], o[b][2], o[b][3]};
          __builtin_amdgcn_raw_buffer_store_b128(v, mrs, par + we_off(ca, cbk, ckp) + (kk * 4 + b) * 1024 + lane * 16, 0, AUX_SYS);
          __builtin_amdgcn_raw_buffer_store_b128(v, mrs, goff + (kk * 4 + b) * 1024, 0, AUX_SYS);
        }
      }
    }
    WAIT_VM(0);
    if (flag && lane < 8) st_sys(lws + flag, call);
  }
  const unsigned ctl = (unsigned)(size_t)(__attribute__((address_space(3))) char*)(lds + 131072 + (wave & 3) * 16);
  auto ctl_st = [&](int slot, int v) { asm volatile("ds_write_b32 %0, %1" :: "v"(ctl + slot * 4), "v"(v) : "memory"); };
  auto ctl_ld = [&](int slot) {
    int v; asm volatile("ds_read_b32 %0, %1\n\ts_waitcnt lgkmcnt(0)" : "=&v"(v) : "v"(ctl + slot * 4) : "memory");
    return __builtin_amdgcn_readfirstlane(v);
  };
  if (wave >= 4 && lane < 4) ctl_st(lane, 0);
  RAW_BARRIER();
  if (wave == 7 && lane == 0) __hip_atomic_store(cnt, call, __ATOMIC_RELAXED, __HIP_MEMORY_SCOPE_AGENT);

  if (wi < 7) {
    const int pair = wave & 3;
    if (pair >= PPX) return;
    const int peer = wi == 0 ? rank ^ 4 : rank ^ xmask(wi - 1);
    const int ntask = wi == 0 ? N_PART : N_OTHER;
    constexpr int cn = 8 * PPX;
    const int cw = T_0 + pair * 8 + xcd;
    char* const pws = p.ws[peer];
    char* const ring = lds + pair * 32768;
    auto decode = [&](int i) {
      Task t; t.src = peer;
      int cp = 0;
      if (i < T_A) {
        const int kq = i / 40, u = i - kq * 40;
        if (u < 24) { const int rq = u / 6; t.type = 1; t.a = 2 + u - 6 * rq; t.b = kq * 4 + rq; return t; }
        t.type = 0; cp = (u - 24) & 7; t.b = 2 * kq + ((u - 24) >> 3);
      } else if (i < T_B) { t.type = 0; cp = 8 + (i & 7); t.b = (i - T_A) >> 3;
      } else if (i < T_X) { t.type = 0; t.b = (i - T_B) / NB; cp = 16 + i - T_B - NB * t.b; }
      else if (wi != 0 ? i < T_O : i >= N_PART - 32) {
        const int u = i - (wi != 0 ? T_X : N_PART - 32);
        t.type = 1; t.a = u & 1; t.b = u >> 1; return t;
      } else if (wi != 0) {
        t.type = 3; t.b = (i - T_O) & 15; cp = (i - T_O) >> 4; t.a = (peer >> 2) ? 31 - cp : cp; return t;
      } else {
        int j = i - T_X, kp, u;
        if (j < LG1 * 56) { kp = j / (LG1 * 7); u = j - kp * (LG1 * 7); }
        else { j -= LG1 * 56; kp = j / ((NF - LG1) * 7); u = j - kp * ((NF - LG1) * 7) + LG1 * 7; }
        const int jb = u / 7, k = u - jb * 7;
        cp = 32 - NF + jb; t.b = kp;
        if (k == 0) t.type = 0; else { t.type = 2; t.src = rank ^ xmask(k - 1); }
      }
      t.a = hf ? 31 - cp : cp;
      return t;
    };
    if (wave >= 4) {
      const __amdgpu_buffer_rsrc_t prs = make_rsrc(pws, CTL);
      int pend = -1;
#pragma unroll 1
      for (int k = 0;; ++k) {
        const int slot = k & 3;
        int c = ctl_ld(slot);
        if (c <= 0 && c != -2) {
          if (pend >= 0) { WAIT_VM(0); ctl_st(pend, -1); pend = -1; }
          while ((c = ctl_ld(slot)) <= 0 && c != -2) __builtin_amdgcn_s_sleep(1);
        }
        if (c == -2) break;
        const unsigned soff = (unsigned)(c >> 1) << 13;
        if (c & 1) {
#pragma unroll
          for (int j = 0; j < 8; ++j)
            __builtin_amdgcn_raw_ptr_buffer_load_lds(prs, LDS3(ring + slot * 8192 + j * 1024), 16, lane * 16, soff + j * 1024, 0, AUX_SYS);
          WAIT_VM(8);
          if (pend >= 0) ctl_st(pend, -1);
        } else {
#pragma unroll
          for (int j = 0; j < 8; ++j) {
            __builtin_amdgcn_raw_ptr_buffer_load_lds(prs, LDS3(ring + slot * 8192 + j * 1024), 16, lane * 16, soff + j * 1024, 0, AUX_SYS);
            WAIT_VM_(PULL_WIN);
            if (j == (PULL_WIN > 0 ? PULL_WIN - 1 : 0) && pend >= 0) ctl_st(pend, -1);
          }
        }
        pend = slot;
      }
      if (pend >= 0) { WAIT_VM(0); ctl_st(pend, -1); }
    } else {
      const unsigned qoff = lane == 0 ? QUE + peer * 128 : 0xfffffff0u;
      const unsigned tbase = (call - 1u) * (unsigned)(ntask - T_0 - (RING_D - 1) * cn) - (unsigned)(T_0 + RING_D * cn);
      bool staged = false, given = false, ended = false, drawing = true, dead = false;
      unsigned tiles = 0;
      unsigned long long todo = 0;
      int ntodo = RING_D;
#pragma unroll
      for (int j = 0; j < RING_D; ++j) todo |= (unsigned long long)(cw + j * cn) << (16 * j);
      unsigned long long held = 0;
      int ka = 0, kf = 0;
      int tk = __builtin_amdgcn_raw_ptr_buffer_atomic_add_i32(1, mrs, qoff, 0, 0);
      unsigned long long t0 = __builtin_amdgcn_s_memrealtime();
      auto ready = [&](unsigned i) {
        if (dead) return true;
        if (!staged) {
          u32x4 f0, f1;
          asm volatile("global_load_dwordx4 %0, %2, off sc0 sc1\n\t"
                       "global_load_dwordx4 %1, %3, off sc0 sc1\n\t"
                       "s_waitcnt vmcnt(0)" : "=&v"(f0), "=&v"(f1)
                       : "v"(myws + SFLAG + peer * 2048 + lane * 16), "v"(myws + SFLAG + peer * 2048 + 1024 + c31 * 16) : "memory");
          if (!(staged = __all(min(min4(f0), min4(f1)) >= call))) {
            if (i >= T_X) return false;
            const Task t = decode((int)i);
            const int w = t.type == 0 ? t.a * 8 + t.b : t.a * 16 + t.b;
            const u32x4 f = t.type == 0 ? f0 : f1;
            const unsigned fw = (w & 2) ? ((w & 1) ? f.w : f.z) : ((w & 1) ? f.y : f.x);
            return (unsigned)__builtin_amdgcn_readlane(fw, w >> 2) >= call;
          }
          if (pair == 0) STAMPW(6);
        }
        if (wi == 0) {
          if (i < T_X || given) return true;
          const bool skip = ((lane & 3) == (rank & 3)) || (hf ? (lane >> 3) >= NF : (lane >> 3) < 8 - NF);
          u32x4 f0, f1;
          asm volatile("global_load_dwordx4 %0, %2, off sc0 sc1\n\t"
                       "global_load_dwordx4 %1, %2, off offset:16 sc0 sc1\n\t"
                       "s_waitcnt vmcnt(0)" : "=&v"(f0), "=&v"(f1) : "v"(myws + GFLAG + lane * 32) : "memory");
          return given = __all(skip || min(min4(f0), min4(f1)) >= call);
        }
        if (i < T_O) return true;
        const unsigned cp = (i - T_O) >> 4, bit = 1u << ((peer >> 2) ? 31 - cp : cp);
        if (tiles & bit) return true;
        unsigned f;
        asm volatile("global_load_dword %0, %1, off sc0 sc1\n\ts_waitcnt vmcnt(0)" : "=&v"(f) : "v"(myws + DONE + peer * 128 + c31 * 4) : "memory");
        tiles = (unsigned)__ballot(f >= call);
        return (tiles & bit) != 0;
      };
      auto assign = [&]() {
        bool any = false;
        while (ntodo > 0) {
          const unsigned nxt = (unsigned)todo & 0xffffu;
          if (nxt >= (unsigned)ntask) { ctl_st(ka & 3, -2); ended = true; ntodo = 0; break; }
          if (!ready(nxt)) break;
          {
            const Task t = decode((int)nxt);
            unsigned soff;
            if (t.type == 0) soff = we_off(rank & 3, t.a, t.b);
            else if (t.type == 1) soff = xe_off(t.a < 2 ? 2 * rank + t.a : 2 * (rank ^ xmask(t.a - 2)) + hf, t.b);
            else if (t.type == 2) soff = ws_off(t.a, t.src, t.b);
            else soff = OS + (unsigned)(((wi - 1) * 32 + t.a) * 16 + t.b) * 8192;
            ctl_st(ka & 3, (int)(((par + soff) >> 12) | (nxt + TAILN >= (unsigned)ntask ? 1u : 0u)));
            const unsigned long long pk = t.type | (t.a << 2) | (t.b << 7) | (t.src << 11);
            held = (held & ~(0xffffull << (16 * (ka & 3)))) | (pk << (16 * (ka & 3)));
          }
          ++ka; any = true;
          todo >>= 16; --ntodo;
        }
        return any;
      };
#pragma unroll 1
      for (unsigned polls = 1;; ++polls) {
        bool progress = assign();
        if (kf < ka && ctl_ld(kf & 3) == -1) {
          const char* l = ring + (kf & 3) * 8192 + lane * 16;
          u32x4 v[8];
#pragma unroll
          for (int j = 0; j < 8; ++j) v[j] = *(const u32x4*)(l + j * 1024);
          const int pk = (int)(held >> (16 * (kf & 3)));
          const Task t = {pk & 3, (pk >> 2) & 31, (pk >> 7) & 15, (pk >> 11) & 7};
          ++kf;
          unsigned flag = 0;
          if (t.type == 3) {
            const unsigned off = (unsigned)((peer >> 2) * 256 + t.b * 16 + 4 * h) * (GN * 4) + ((peer & 3) * 4096 + t.a * 128) * 4 + c31 * 16;
            const unsigned poison = dead ? 0x7fc00000u : 0u;
#pragma unroll
            for (int j = 0; j < 8; ++j)
              __builtin_amdgcn_raw_buffer_store_b128(v[j] | poison, ors, off + ((j & 3) + 8 * (j >> 2)) * (GN * 4), 0, OUT_AUX);
          } else {
            unsigned off, s1, s2;
            if (t.type == 1) {
              off = xs_off(t.a, peer, t.b); s1 = 8192; s2 = 131072;
              flag = XFLAG + ((t.a * 8 + peer) * 16 + t.b) * 4;
            } else {
              off = ws_off(t.a, t.src, t.b); s1 = 2048; s2 = 4096;
              flag = WFLAG + ((t.a * 8 + t.src) * 8 + t.b) * 4;
            }
            off += par + lane * 16;
#pragma unroll
            for (int j = 0; j < 8; ++j)
              __builtin_amdgcn_raw_buffer_store_b128(v[j], mrs, off + (j >> 2) * s2 + ((j >> 1) & 1) * s1 + (j & 1) * 1024, 0, AUX_SYS);
          }
          WAIT_VM(0);
          {
            const unsigned got = min((unsigned)__builtin_amdgcn_readfirstlane(tk) - tbase, 0xffffu);
            if (drawing) { todo |= (unsigned long long)got << (16 * ntodo); ++ntodo; drawing = got < (unsigned)ntask; }
            tk = __builtin_amdgcn_raw_ptr_buffer_atomic_add_i32(1, mrs, drawing ? qoff : 0xfffffff0u, 0, 0);
          }
          if (flag) {
            if (lane == 0) st_sys(myws + flag, call);
            if (lane == 1 && wi != 0 && t.type == 0 && (hf ? t.a >= 32 - NF : t.a < NF))
              st_sys(p.ws[rank ^ 4] + GFLAG + (((t.a & 7) * 8 + peer) * 8 + t.b) * 4, call);
          }
          progress = true;
        }
        if (ended && kf == ka) break;
        if (progress) { polls = 0; t0 = __builtin_amdgcn_s_memrealtime(); }
        else {
          if ((polls & 63) == 0 && give_up(tmo, call, t0)) dead = true;
          __builtin_amdgcn_s_sleep(ST_SLEEP);
        }
      }
      STAMPW(8 + wave);
    }
    return;
  }

  const int njob = (wi >= 8 && wi < 15) ? 2 : 1;
#pragma unroll 1
  for (int job = 0; job < njob; ++job) {
    int lrb, jc;
    if (wi == 7 || job) { lrb = wi & 1; jc = (wi >> 1) & 3; }
    else { jc = (wi - 8) / 6; lrb = 2 + (wi - 8) - 6 * jc; }
    const int cp = xcd + 8 * jc;
    const int cb = hf ? 31 - cp : cp;
    auto stage_id = [&](int q) {
      if (q < 8) return rank * 8 + q;
      q -= 8;
      if (lrb >= 2) { const int kp = q / 7, k = q - 7 * kp; return (rank ^ (k == 0 ? 4 : xmask(k - 1))) * 8 + kp; }
      if (q >= 48) return (rank ^ 4) * 8 + q - 48;
      const int kp = q / 6;
      return (rank ^ xmask(q - 6 * kp)) * 8 + kp;
    };
    const int my_id = stage_id(lane);
    const unsigned* const xf = (const unsigned*)(myws + XFLAG) + lrb * 128 + (my_id >> 1) * 4;
    const unsigned* const wf = (const unsigned*)(myws + WFLAG) + cb * 64 + my_id;
    int ready = 0;
    auto ensure = [&](int q) {
      if (ready > q) return;
      const unsigned long long t0 = __builtin_amdgcn_s_memrealtime();
      unsigned polls = 0;
      for (;;) {
        u32x4 fx; unsigned fw;
        asm volatile("global_load_dwordx4 %0, %2, off sc0 sc1\n\t"
                     "global_load_dword %1, %3, off sc0 sc1\n\t"
                     "s_waitcnt vmcnt(0)"
                     : "=&v"(fx), "=&v"(fw) : "v"(xf), "v"(wf) : "memory");
        const unsigned long long m = __ballot(min(min4(fx), fw) >= call);
        ready = (~m == 0ull) ? 64 : __builtin_ctzll(~m);
        if (ready > q) return;
        if ((++polls & 15) == 0 && give_up(tmo, call, t0)) { ready = 64; return; }
        __builtin_amdgcn_s_sleep(POLL_SLEEP);
      }
    };
    const unsigned a_base = par + (unsigned)lrb * (1u << 20) + wave * 2048;
    const unsigned b_base = par + XS_BYTES + (unsigned)cb * (512u << 10) + wave * 1024;
    auto issue = [&](int q, int slot) {
      const int sid = stage_id(q), id = (sid & 7) * 8 + (sid >> 3);
      char* l = lds + slot * STAGE;
      auto* l0 = LDS3(l + wave * 2048);
      auto* l1 = LDS3(l + wave * 2048 + 1024);
      auto* l2 = LDS3(l + 16384 + wave * 1024);
      __builtin_amdgcn_raw_ptr_buffer_load_lds(mrs, l0, 16, lane * 16, a_base + id * 16384, 0, AUX_SYS);
      __builtin_amdgcn_raw_ptr_buffer_load_lds(mrs, l1, 16, lane * 16, a_base + id * 16384 + 1024, 0, AUX_SYS);
      __builtin_amdgcn_raw_ptr_buffer_load_lds(mrs, l2, 16, lane * 16, b_base + id * 8192, 0, AUX_SYS);
    };
    i32x16 acc[4];
#pragma unroll
    for (int b = 0; b < 4; ++b)
#pragma unroll
      for (int i = 0; i < 16; ++i) acc[b][i] = 0;
    i32x4 fa[2][2], fb[2][2][4];
    auto rd = [&](int buf, int slot) {
      const char* l = lds + slot * STAGE + lane * 16;
#pragma unroll
      for (int ks = 0; ks < 2; ++ks) {
        fa[buf][ks] = *(const i32x4*)(l + ks * 8192 + wave * 1024);
#pragma unroll
        for (int b = 0; b < 4; ++b) fb[buf][ks][b] = *(const i32x4*)(l + 16384 + ks * 4096 + b * 1024);
      }
    };
    auto mm = [&](int buf) {
#pragma unroll
      for (int ks = 0; ks < 2; ++ks)
#pragma unroll
        for (int b = 0; b < 4; ++b) acc[b] = __builtin_amdgcn_mfma_i32_32x32x32_i8(fa[buf][ks], fb[buf][ks][b], acc[b], 0, 0, 0);
    };
    auto nx = [&](int s) { return s == NSLOT - 1 ? 0 : s + 1; };

    if (wave == 0) ensure(NSLOT - 1);
    RAW_BARRIER();
    STAMP(1);
#pragma unroll
    for (int q = 0; q < NSLOT; ++q) issue(q, q);
    WAIT_VM(15);
    RAW_BARRIER();
    rd(0, 0);
    int s0 = 0;
#pragma unroll 1
    for (int i = 0; i < NSTAGE - NSLOT; i += 2) {
      if (wave == 0) ensure(i + NSLOT);
      WAIT_VM(12); RAW_BARRIER();
      issue(i + NSLOT, s0); rd(1, nx(s0)); mm(0);
      s0 = nx(s0);
      if (wave == 0) ensure(i + NSLOT + 1);
      WAIT_VM(12); RAW_BARRIER();
      issue(i + NSLOT + 1, s0); rd(0, nx(s0)); mm(1);
      s0 = nx(s0);
    }
    STAMP(2);
    WAIT_VM(12); RAW_BARRIER(); rd(1, nx(s0)); mm(0); s0 = nx(s0);
    WAIT_VM(9);  RAW_BARRIER(); rd(0, nx(s0)); mm(1); s0 = nx(s0);
    WAIT_VM(6);  RAW_BARRIER(); rd(1, nx(s0)); mm(0); s0 = nx(s0);
    WAIT_VM(3);  RAW_BARRIER(); rd(0, nx(s0)); mm(1); s0 = nx(s0);
    WAIT_VM(0);  RAW_BARRIER(); rd(1, nx(s0)); mm(0);
    mm(1);
    STAMP(3);

    {
      const float scale = p.sx[0] * p.sw[0];
      const unsigned poison = timed_out(tmo, call) ? 0x7fc00000u : 0u;
      const unsigned ooff = (unsigned)(lrb * 256 + wave * 32 + 4 * h) * (GN * 4) + ((rank & 3) * 4096 + cb * 128) * 4 + c31 * 16;
      const unsigned soff = par + OS + (unsigned)(((lrb - 2) * 32 + cb) * 8 + wave) * 16384 + lane * 16;
#pragma unroll
      for (int i = 0; i < 16; ++i) {
        u32x4 o;
#pragma unroll
        for (int b = 0; b < 4; ++b) o[b] = __builtin_bit_cast(unsigned, silu_dq(acc[b][i], scale)) | poison;
        if (lrb < 2) __builtin_amdgcn_raw_buffer_store_b128(o, ors, ooff + ((i & 3) + 8 * (i >> 2)) * (GN * 4), 0, OUT_AUX);
        else __builtin_amdgcn_raw_buffer_store_b128(o, mrs, soff + i * 1024, 0, AUX_SYS);
      }
    }
    WAIT_VM(0);
    RAW_BARRIER();
    STAMP(4);
    if (wave == 0 && lane == 0 && lrb >= 2) st_sys(p.ws[rank ^ xmask(lrb - 2)] + DONE + (rank * 32 + cb) * 4, call);
  }
}

}

extern "C" void kernel_launch(int rank, int nranks, ncclComm_t comm,
                              hipStream_t stream,
                              void* const* d_in, const int* in_sizes, int n_in,
                              void* d_out, int out_size,
                              const int* in_dtypes, int out_dtype,
                              void* const* d_out_all,
                              void* const* d_ws_all, size_t ws_bytes) {
  (void)comm; (void)in_sizes; (void)n_in; (void)d_out_all; (void)out_size; (void)in_dtypes; (void)out_dtype;
  if (nranks != R || (unsigned)rank >= (unsigned)R || ws_bytes < WS_USED) {
    fprintf(stderr, "unsupported configuration\n");
    return;
  }
  Params p;
  p.x = (const int8_t*)d_in[0];
  p.w = (const int8_t*)d_in[1];
  p.sx = (const float*)d_in[2];
  p.sw = (const float*)d_in[3];
  p.out = (float*)d_out;
  for (int i = 0; i < R; ++i) p.ws[i] = (char*)d_ws_all[i];
  p.rank = rank; p.pad = 0;
  p.myws = p.ws[rank];
  fused_kernel<<<NWG, NTHR, 0, stream>>>(p);
  HIP_CHECK(hipGetLastError());
}
```

```cpp
#include <hip/hip_runtime.h>
#include <rccl/rccl.h>
#include <cstdio>
#include <cstdint>

#define HIP_CHECK(expr) do { hipError_t _e = (expr); if (_e != hipSuccess) { \
    fprintf(stderr, "HIP error %s at %s:%d\n", hipGetErrorString(_e), __FILE__, __LINE__); \
    return; } } while (0)

#define PULL_WIN 2
#define RING_D 3
#define OUT_AUX 17
#define ST_SLEEP 2
#define STG_DELAY 20
#define PUSH_WIN 2
#define TAILN 8
#define PUSHN 20
#define MINI_DELAY 3
#define POLL_SLEEP 4
#define NF 7
#define LG1 4

namespace {

typedef int      i32x4  __attribute__((ext_vector_type(4)));
typedef int      i32x16 __attribute__((ext_vector_type(16)));
typedef unsigned u32x4  __attribute__((ext_vector_type(4)));

constexpr int R = 8;
constexpr int GN = 16384;
constexpr int KS = 512;
constexpr int NWG = 256, NTHR = 512;

constexpr unsigned XS_BYTES = 8u << 20;
constexpr unsigned WS_BYTES = 16u << 20;
constexpr unsigned XE = XS_BYTES + WS_BYTES;
constexpr unsigned WE = XE + (2u << 20);
constexpr unsigned OS = WE + (8u << 20);
constexpr unsigned PS = OS + (24u << 20);
constexpr unsigned BUF_BYTES = PS + (24u << 20);
constexpr unsigned CTL = 2 * BUF_BYTES;
constexpr unsigned XFLAG = CTL;
constexpr unsigned WFLAG = XFLAG + 8 * 128 * 4;
constexpr unsigned DONE = WFLAG + 32 * 64 * 4;
constexpr unsigned SFLAG = DONE + 1024;
constexpr unsigned GFLAG = SFLAG + 16384;
constexpr unsigned EFLAG = GFLAG + 2048;
constexpr unsigned PFLAG = EFLAG + 4096;
constexpr unsigned CNT = PFLAG + 1024;
constexpr unsigned TMO = CNT + 1024;
constexpr unsigned QUE = TMO + 1024;
constexpr unsigned DBG = QUE + 1024;
constexpr unsigned WS_USED = DBG + 2 * 256 * 16 * 8;

constexpr int NB = 16 - NF, T_0 = PUSHN  , T_A = 160, T_B = 224, T_X = T_B + 8 * NB, T_O = T_X + 32, N_OTHER = T_O + 512, N_PART = T_O + 56 * NF;
constexpr int STAGE = 24576, NSLOT = 6;
constexpr unsigned long long DEADLINE = 50000000ull;
constexpr int AUX_SYS = 17;

struct Params {
  const int8_t* x; const int8_t* w; char* myws; int rank; int pad;
  const float* sx; const float* sw;
  float* out; char* ws[R];
};

#define WAIT_VM(n) asm volatile("s_waitcnt vmcnt(" #n ")" ::: "memory")
#define WAIT_VM_(n) WAIT_VM(n)
#define RAW_BARRIER() do { asm volatile("s_waitcnt lgkmcnt(0)" ::: "memory"); __builtin_amdgcn_s_barrier(); asm volatile("" ::: "memory"); } while (0)
#define LDS3(p) ((__attribute__((address_space(3))) void*)(p))

#define STAMP(i) do {} while (0)
#define STAMPW(i) do {} while (0)

__device__ __forceinline__ __amdgpu_buffer_rsrc_t make_rsrc(const void* p, unsigned bytes) {
  return __builtin_amdgcn_make_buffer_rsrc((void*)p, 0, bytes, 0x00020000);
}
__device__ __forceinline__ void st_sys(void* p, unsigned v) {
  __hip_atomic_store((unsigned*)p, v, __ATOMIC_RELAXED, __HIP_MEMORY_SCOPE_SYSTEM);
}
__device__ __forceinline__ bool timed_out(unsigned* tmo, unsigned call) {
  return __builtin_amdgcn_readfirstlane(__hip_atomic_load(tmo, __ATOMIC_RELAXED, __HIP_MEMORY_SCOPE_AGENT)) == call;
}
__device__ __forceinline__ bool give_up(unsigned* tmo, unsigned call, unsigned long long t0) {
  if (timed_out(tmo, call) || __builtin_amdgcn_s_memrealtime() - t0 > DEADLINE) {
    __hip_atomic_store(tmo, call, __ATOMIC_RELAXED, __HIP_MEMORY_SCOPE_AGENT);
    return true;
  }
  return false;
}
__device__ __forceinline__ int xmask(int j) { return j + 1 + (j >= 3); }
__device__ __forceinline__ unsigned min4(u32x4 f) { return min(min(f.x, f.y), min(f.z, f.w)); }

__device__ __forceinline__ void transpose4(unsigned d0, unsigned d1, unsigned d2, unsigned d3,
                                           unsigned& o0, unsigned& o1, unsigned& o2, unsigned& o3) {
  unsigned t0 = __builtin_amdgcn_perm(d1, d0, 0x05010400u);
  unsigned t1 = __builtin_amdgcn_perm(d1, d0, 0x07030602u);
  unsigned t2 = __builtin_amdgcn_perm(d3, d2, 0x05010400u);
  unsigned t3 = __builtin_amdgcn_perm(d3, d2, 0x07030602u);
  o0 = __builtin_amdgcn_perm(t2, t0, 0x05040100u);
  o1 = __builtin_amdgcn_perm(t2, t0, 0x07060302u);
  o2 = __builtin_amdgcn_perm(t3, t1, 0x05040100u);
  o3 = __builtin_amdgcn_perm(t3, t1, 0x07060302u);
}

__device__ __forceinline__ float silu_dq(int acc, float s) {
  float y = (float)acc * s;
  float c = __builtin_amdgcn_fmed3f(y, -60.0f, 60.0f);
  float e = __builtin_amdgcn_exp2f(c * -1.44269504088896341f);
  return y * __builtin_amdgcn_rcpf(1.0f + e);
}

__device__ __forceinline__ unsigned xs_off(int lrb, int src, int b) { return (unsigned)(lrb * 64 + (b >> 2) * 16 + src) * 16384 + (b & 3) * 2048; }
__device__ __forceinline__ unsigned xs_piece(int j) { return (j >> 2) * 131072 + ((j >> 1) & 1) * 8192 + (j & 1) * 1024; }
__device__ __forceinline__ unsigned ws_off(int cb, int src, int kp) { return XS_BYTES + (unsigned)(cb * 64 + kp * 8 + src) * 8192; }
__device__ __forceinline__ unsigned xe_off(int rb, int b) { return XE + (unsigned)((b * 2 + (rb & 1)) * 8 + (rb >> 1)) * 8192; }
__device__ __forceinline__ unsigned we_off(int q, int cb, int kp) { return WE + (unsigned)((q * 8 + kp) * 32 + cb) * 8192; }

struct Task { int type, a, b, src; };

__global__ void __launch_bounds__(NTHR, 2) fused_kernel(Params p) {
  __shared__ __attribute__((aligned(16))) char lds[STAGE * NSLOT];
  const int lane = threadIdx.x & 63;
  const int wave = __builtin_amdgcn_readfirstlane(threadIdx.x >> 6);
  const int bid = blockIdx.x;
  const int rank = p.rank, hf = rank >> 2;
  char* const myws = p.myws;
  const int8_t* const px = p.x;
  const int8_t* const pw = p.w;
  asm volatile("" :: "s"(px), "s"(pw), "s"(myws), "s"(rank));
  unsigned* const tmo = (unsigned*)(myws + TMO);
  unsigned* const cnt = (unsigned*)(myws + CNT) + bid;
  const unsigned cnt_v = __hip_atomic_load(cnt, __ATOMIC_RELAXED, __HIP_MEMORY_SCOPE_AGENT);
  const int h = lane >> 5, c31 = lane & 31;
  const int xcd = bid & 7, wi = bid >> 3;
  const __amdgpu_buffer_rsrc_t mrs = make_rsrc(myws, WS_USED);
  const __amdgpu_buffer_rsrc_t ors = make_rsrc(p.out, 512u * GN * 4u);

  char* const lws = p.ws[lane & 7];
  const int n = bid - 56, sc = (wave - 1) * 200 + n;
  const int pu = n >> 3, pd = (n + pu) & 7, pm = pd ^ rank;
  const int prq = pm == 4 ? pu / 6 : pu / 5, pv = pu - (pm == 4 ? 6 : 5) * prq;
  const int plrb = 2 + pv + (pm != 4 && pv >= pm - 1 - (pm > 4));
  const bool active = n >= 0 && (wave == 0 ? (n < 8 * T_0 && pd != rank) : sc < 1280);
  int kind = 0, ca = 0, cbk = 0, ckp = 0;
  if (wave > 0) {
    const int kq = sc / 192, u = sc < 768 ? sc - kq * 192 : sc - 768;
    if (sc < 768 && u >= 128) { kind = 1; ca = u & 15; cbk = kq * 4 + ((u - 128) >> 4); }
    else {
      kind = 2; ca = (u >> 4) & 3;
      if (sc < 768) { ckp = 2 * kq + (u >> 6); cbk = (u & 15) + (u & 8) * 2; } else { ckp = u >> 6; cbk = 8 + (u & 15); }
    }
  }
  if (active) {
    const int r8 = lane >> 3, k8 = lane & 7;
    const char* src = kind == 0 ? (const char*)px + (size_t)(512 * (pd ^ xmask(plrb - 2)) + 256 * (pd >> 2) + 64 * prq) * KS
                    : kind == 1 ? (const char*)px + (size_t)(ca * 256 + (cbk & 3) * 64) * KS + (cbk >> 2) * 128
                                : (const char*)pw + (size_t)(64 * ckp) * GN + ca * 4096 + cbk * 128;
    const unsigned stride = kind < 2 ? KS : GN;
    const __amdgpu_buffer_rsrc_t srs = make_rsrc(src, 63 * stride + 128);
    const unsigned voff = r8 * stride + ((kind < 2 ? k8 ^ r8 : k8) * 16);
    for (int d = 1; d < wave; ++d) __builtin_amdgcn_s_sleep(STG_DELAY);
#pragma unroll
    for (int j = 0; j < 8; ++j)
      __builtin_amdgcn_raw_ptr_buffer_load_lds(srs, LDS3(lds + wave * 8192 + j * 1024), 16, voff, j * 8 * stride, 0, 0);
  }
  asm volatile("" ::: "memory");
  const unsigned call = __builtin_amdgcn_readfirstlane(cnt_v) + 1u;
  const unsigned par = (call & 1u) * BUF_BYTES;
  if (active) {
    WAIT_VM(0);
    const char* l = lds + wave * 8192;
    unsigned flag = 0;
    if (kind == 1) {
      const int rb = ca, b = cbk;
      auto lrb_of = [&](int d) { const int m = (rb >> 1) ^ d; return m == 0 ? (rb & 1) : (((m & 3) && (rb & 1) == (d >> 2)) ? 1 + m - (m > 4) : -1); };
      const int lrb = lrb_of(rank), lrbl = lrb_of(lane & 7);
      const unsigned goff = lrb >= 0 ? par + xs_off(lrb, rank, b) + lane * 16 : 0xf0000000u;
      if (lrbl >= 0) flag = lane == rank ? XFLAG + ((lrbl * 8 + rank) * 16 + b) * 4 : SFLAG + rank * 2048 + 1024 + (lrbl * 16 + b) * 4;
#pragma unroll
      for (int j = 0; j < 8; ++j) {
        const u32x4 v = *(const u32x4*)(l + c31 * 128 + (((2 * (j >> 1) + h) ^ (c31 & 7)) * 16) + (j & 1) * 4096);
        __builtin_amdgcn_raw_buffer_store_b128(v, mrs, par + xe_off(rb, b) + j * 1024 + lane * 16, 0, AUX_SYS);
        __builtin_amdgcn_raw_buffer_store_b128(v, mrs, goff + xs_piece(j), 0, AUX_SYS);
      }
    } else if (kind == 0) {
      const __amdgpu_buffer_rsrc_t drs = make_rsrc(p.ws[pd], CTL);
      const unsigned goff = par + xs_off(plrb, rank, prq) + lane * 16;
      if (lane == pd) flag = XFLAG + ((plrb * 8 + rank) * 16 + prq) * 4;
#pragma unroll
      for (int j = 0; j < 8; ++j) {
        const u32x4 v = *(const u32x4*)(l + c31 * 128 + (((2 * (j >> 1) + h) ^ (c31 & 7)) * 16) + (j & 1) * 4096);
        __builtin_amdgcn_raw_buffer_store_b128(v, drs, goff + xs_piece(j), 0, AUX_SYS);
        WAIT_VM_(PUSH_WIN);
      }
    } else {
      const unsigned goff = ca == (rank & 3) ? par + ws_off(cbk, rank, ckp) + lane * 16 : 0xf0000000u;
      if ((lane & 3) == ca) flag = lane == rank ? WFLAG + ((cbk * 8 + rank) * 8 + ckp) * 4 : SFLAG + rank * 2048 + (cbk * 8 + ckp) * 4;
      else if (lane == rank) flag = EFLAG + ((ca * 32 + cbk) * 8 + ckp) * 4;
#pragma unroll
      for (int kk = 0; kk < 2; ++kk) {
        unsigned o[4][4];
#pragma unroll
        for (int q = 0; q < 4; ++q) {
          const unsigned* lp = (const unsigned*)(l + (kk * 32 + h * 16 + q * 4) * 128 + c31 * 4);
          transpose4(lp[0], lp[32], lp[64], lp[96], o[0][q], o[1][q], o[2][q], o[3][q]);
        }
#pragma unroll
        for (int b = 0; b < 4; ++b) {
          const u32x4 v = {o[b][0], o[b][1], o[b][2], o[b][3]};
          __builtin_amdgcn_raw_buffer_store_b128(v, mrs, par + we_off(ca, cbk, ckp) + (kk * 4 + b) * 1024 + lane * 16, 0, AUX_SYS);
          __builtin_amdgcn_raw_buffer_store_b128(v, mrs, goff + (kk * 4 + b) * 1024, 0, AUX_SYS);
        }
      }
    }
    WAIT_VM(0);
    if (flag && lane < 8) st_sys(lws + flag, call);
  }
  const unsigned ctl = (unsigned)(size_t)(__attribute__((address_space(3))) char*)(lds + 131072 + (wave & 1) * 32);
  auto ctl_st = [&](int slot, int v) { asm volatile("ds_write_b32 %0, %1" :: "v"(ctl + slot * 4), "v"(v) : "memory"); };
  auto ctl_ld = [&](int slot) {
    int v; asm volatile("ds_read_b32 %0, %1\n\ts_waitcnt lgkmcnt(0)" : "=&v"(v) : "v"(ctl + slot * 4) : "memory");
    return __builtin_amdgcn_readfirstlane(v);
  };
  if (wave >= 6 && lane < 8) ctl_st(lane, lane < 4 ? 0 : -1);
  RAW_BARRIER();
  if (wave == 7 && lane == 0) __hip_atomic_store(cnt, call, __ATOMIC_RELAXED, __HIP_MEMORY_SCOPE_AGENT);

  if (wi < 7) {
    const int pair = wave & 1;
    const int peer = wi == 0 ? rank ^ 4 : rank ^ xmask(wi - 1);
    const int ntask = wi == 0 ? N_PART : N_OTHER;
    constexpr int cn = 16;
    const int i0 = (wi == 0 ? 0 : 16) + T_0;
    const int cw = i0 + pair * 8 + xcd;
    char* const pws = p.ws[peer];
    char* const ring = lds + pair * 32768;
    auto decode = [&](int i) {
      Task t; t.src = peer;
      int cp = 0;
      if (i < T_A) {
        int kq, u;
        if (wi == 0) {
          kq = i / 40; u = i - kq * 40;
          if (u < 24) { const int rq = u / 6; t.type = 1; t.a = 2 + u - 6 * rq; t.b = kq * 4 + rq; return t; }
          u -= 24;
        } else {
          kq = (i - 16) / 36; u = i - 16 - kq * 36;
          if (u < 20) { const int rq = u / 5, v = u - 5 * rq; t.type = 1; t.a = 2 + v + (v >= wi - 1); t.b = kq * 4 + rq; return t; }
          u -= 20;
        }
        t.type = 0; cp = u & 7; t.b = 2 * kq + (u >> 3);
      } else if (i < T_B) { t.type = 0; cp = 8 + (i & 7); t.b = (i - T_A) >> 3;
      } else if (i < T_X) { t.type = 0; t.b = (i - T_B) / NB; cp = 16 + i - T_B - NB * t.b; }
      else if (wi != 0 ? i < T_O : i >= N_PART - 32) {
        const int u = i - (wi != 0 ? T_X : N_PART - 32);
        t.type = 1; t.a = u & 1; t.b = u >> 1; return t;
      } else if (wi != 0) {
        t.type = 3; t.b = (i - T_O) & 15; cp = (i - T_O) >> 4; t.a = (peer >> 2) ? 31 - cp : cp; return t;
      } else {
        int j = i - T_X, kp, u;
        if (j < LG1 * 56) { kp = j / (LG1 * 7); u = j - kp * (LG1 * 7); }
        else { j -= LG1 * 56; kp = j / ((NF - LG1) * 7); u = j - kp * ((NF - LG1) * 7) + LG1 * 7; }
        const int jb = u / 7, k = u - jb * 7;
        cp = 32 - NF + jb; t.b = kp;
        if (k == 0) t.type = 0; else { t.type = 2; t.src = rank ^ xmask(k - 1); }
      }
      t.a = hf ? 31 - cp : cp;
      return t;
    };
    if (wave & 2) {
      if (wi == 0) return;
      const float scale = p.sx[0] * p.sw[0];
#pragma unroll 1
      for (int k = wave >> 2;; k += 2) {
        const int slot = k & 3;
        int a;
        while (((a = ctl_ld(4 + slot)) >> 16) < k) __builtin_amdgcn_s_sleep(4);
        if (a & 0x8000) break;
        if ((a >> 16) > k || (a & 3) != 3) continue;
        const int cb = (a >> 2) & 31, part = (a >> 7) & 15;
        const unsigned poff = par + PS + (unsigned)(((wi - 1) * 32 + cb) * 16 + part) * 8192 + lane * 16;
        u32x4 v[8];
#pragma unroll
        for (int j = 0; j < 8; ++j) v[j] = __builtin_amdgcn_raw_buffer_load_b128(mrs, poff + j * 1024, 0, AUX_SYS);
        while (ctl_ld(slot) != -1) __builtin_amdgcn_s_sleep(2);
#pragma unroll
        for (int j = 0; j < 8; ++j) v[j] += *(const u32x4*)(ring + slot * 8192 + lane * 16 + j * 1024);
        asm volatile("" : "+v"(v[0]), "+v"(v[1]), "+v"(v[2]), "+v"(v[3]), "+v"(v[4]), "+v"(v[5]), "+v"(v[6]), "+v"(v[7]) :: "memory");
        ctl_st(slot, -3);
        const unsigned off = (unsigned)((peer >> 2) * 256 + part * 16 + 4 * h) * (GN * 4) + ((peer & 3) * 4096 + cb * 128) * 4 + c31 * 16;
        const unsigned poison = (a & 0x4000) ? 0x7fc00000u : 0u;
#pragma unroll
        for (int j = 0; j < 8; ++j) {
          u32x4 o;
#pragma unroll
          for (int b = 0; b < 4; ++b) o[b] = __builtin_bit_cast(unsigned, silu_dq((int)v[j][b], scale)) | poison;
          __builtin_amdgcn_raw_buffer_store_b128(o, ors, off + ((j & 3) + 8 * (j >> 2)) * (GN * 4), 0, OUT_AUX);
        }
      }
      STAMPW(8 + (wave >> 2));
    } else if (wave >= 4) {
      const __amdgpu_buffer_rsrc_t prs = make_rsrc(pws, CTL);
      int pend = -1;
#pragma unroll 1
      for (int k = 0;; ++k) {
        const int slot = k & 3;
        int c = ctl_ld(slot);
        if (c <= 0 && c != -2) {
          if (pend >= 0) { WAIT_VM(0); ctl_st(pend, -1); pend = -1; }
          while ((c = ctl_ld(slot)) <= 0 && c != -2) __builtin_amdgcn_s_sleep(1);
        }
        if (c == -2) break;
        const unsigned soff = (unsigned)(c >> 1) << 13;
        if (c & 1) {
#pragma unroll
          for (int j = 0; j < 8; ++j)
            __builtin_amdgcn_raw_ptr_buffer_load_lds(prs, LDS3(ring + slot * 8192 + j * 1024), 16, lane * 16, soff + j * 1024, 0, AUX_SYS);
          WAIT_VM(8);
          if (pend >= 0) ctl_st(pend, -1);
        } else {
#pragma unroll
          for (int j = 0; j < 8; ++j) {
            __builtin_amdgcn_raw_ptr_buffer_load_lds(prs, LDS3(ring + slot * 8192 + j * 1024), 16, lane * 16, soff + j * 1024, 0, AUX_SYS);
            WAIT_VM_(PULL_WIN);
            if (j == (PULL_WIN > 0 ? PULL_WIN - 1 : 0) && pend >= 0) ctl_st(pend, -1);
          }
        }
        pend = slot;
      }
      if (pend >= 0) { WAIT_VM(0); ctl_st(pend, -1); }
    } else {
      const unsigned qoff = lane == 0 ? QUE + peer * 128 : 0xfffffff0u;
      const unsigned tbase = (call - 1u) * (unsigned)(ntask - i0 - (RING_D - 1) * cn) - (unsigned)(i0 + RING_D * cn);
      bool staged = false, given = false, ended = false, drawing = true, dead = false;
      unsigned tiles = 0;
      u32x4 sf0 = {0, 0, 0, 0}, sf1 = {0, 0, 0, 0};
      unsigned long long todo = 0;
      int ntodo = RING_D;
#pragma unroll
      for (int j = 0; j < RING_D; ++j) todo |= (unsigned long long)(cw + j * cn) << (16 * j);
      unsigned long long held = 0;
      int ka = 0, kf = 0;
      int tk = __builtin_amdgcn_raw_ptr_buffer_atomic_add_i32(1, mrs, qoff, 0, 0);
      unsigned long long t0 = __builtin_amdgcn_s_memrealtime();
      auto ready = [&](unsigned i) {
        if (dead) return true;
        if (!staged) {
          const Task t = decode((int)min(i, (unsigned)T_X - 1u));
          const int w = t.type == 0 ? t.a * 8 + t.b : t.a * 16 + t.b;
          for (int pass = 0;; ++pass) {
            const u32x4 f = t.type == 0 ? sf0 : sf1;
            const unsigned fw = (w & 2) ? ((w & 1) ? f.w : f.z) : ((w & 1) ? f.y : f.x);
            if (i < T_X && (unsigned)__builtin_amdgcn_readlane(fw, w >> 2) >= call) return true;
            if (pass) return false;
            asm volatile("global_load_dwordx4 %0, %2, off sc0 sc1\n\t"
                         "global_load_dwordx4 %1, %3, off sc0 sc1\n\t"
                         "s_waitcnt vmcnt(0)" : "=&v"(sf0), "=&v"(sf1)
                         : "v"(myws + SFLAG + peer * 2048 + lane * 16), "v"(myws + SFLAG + peer * 2048 + 1024 + c31 * 16) : "memory");
            if ((staged = __all(min(min4(sf0), min4(sf1)) >= call))) break;
          }
          if (pair == 0) STAMPW(6);
        }
        if (wi == 0) {
          if (i < T_X || given) return true;
          const bool skip = ((lane & 3) == (rank & 3)) || (hf ? (lane >> 3) >= NF : (lane >> 3) < 8 - NF);
          u32x4 f0, f1;
          asm volatile("global_load_dwordx4 %0, %2, off sc0 sc1\n\t"
                       "global_load_dwordx4 %1, %2, off offset:16 sc0 sc1\n\t"
                       "s_waitcnt vmcnt(0)" : "=&v"(f0), "=&v"(f1) : "v"(myws + GFLAG + lane * 32) : "memory");
          return given = __all(skip || min(min4(f0), min4(f1)) >= call);
        }
        if (i < T_O) return true;
        const unsigned cp = (i - T_O) >> 4, bit = 1u << ((peer >> 2) ? 31 - cp : cp);
        if (tiles & bit) return true;
        unsigned f;
        asm volatile("global_load_dword %0, %1, off sc0 sc1\n\ts_waitcnt vmcnt(0)" : "=&v"(f)
                     : "v"(myws + (h ? PFLAG + (wi - 1) * 128 : DONE + peer * 128) + c31 * 4) : "memory");
        const unsigned long long m = __ballot(f >= call);
        tiles = (unsigned)m & (unsigned)(m >> 32);
        return (tiles & bit) != 0;
      };
      auto assign = [&]() {
        bool any = false;
        while (ntodo > 0) {
          const unsigned nxt = (unsigned)todo & 0xffffu;
          if (nxt >= (unsigned)ntask) { ctl_st(ka & 3, -2); ended = true; ntodo = 0; break; }
          if (!ready(nxt)) break;
          {
            const Task t = decode((int)nxt);
            unsigned soff;
            if (t.type == 0) soff = we_off(rank & 3, t.a, t.b);
            else if (t.type == 1) soff = xe_off(t.a < 2 ? 2 * rank + t.a : 2 * (rank ^ xmask(t.a - 2)) + hf, t.b);
            else if (t.type == 2) soff = ws_off(t.a, t.src, t.b);
            else soff = OS + (unsigned)(((wi - 1) * 32 + t.a) * 16 + t.b) * 8192;
            const unsigned pk = t.type | (t.a << 2) | (t.b << 7) | (t.src << 11) | (dead ? 0x4000 : 0);
            ctl_st(ka & 3, (int)(((par + soff) >> 12) | (nxt + TAILN >= (unsigned)ntask ? 1u : 0u)));
            ctl_st(4 + (ka & 3), (int)((ka << 16) | pk));
            held = (held & ~(0xffffull << (16 * (ka & 3)))) | ((unsigned long long)pk << (16 * (ka & 3)));
          }
          ++ka; any = true;
          todo >>= 16; --ntodo;
        }
        return any;
      };
#pragma unroll 1
      for (unsigned polls = 1;; ++polls) {
        bool progress = assign();
        const int pk = (int)(held >> (16 * (kf & 3)));
        if (kf < ka && ctl_ld(kf & 3) == ((pk & 3) == 3 ? -3 : -1)) {
          const char* l = ring + (kf & 3) * 8192 + lane * 16;
          const Task t = {pk & 3, (pk >> 2) & 31, (pk >> 7) & 15, (pk >> 11) & 7};
          ++kf;
          unsigned flag = 0;
          if (t.type != 3) {
            unsigned off, s1, s2;
            if (t.type == 1) {
              off = xs_off(t.a, peer, t.b); s1 = 8192; s2 = 131072;
              flag = XFLAG + ((t.a * 8 + peer) * 16 + t.b) * 4;
            } else {
              off = ws_off(t.a, t.src, t.b); s1 = 2048; s2 = 4096;
              flag = WFLAG + ((t.a * 8 + t.src) * 8 + t.b) * 4;
            }
            off += par + lane * 16;
#pragma unroll
            for (int j = 0; j < 8; ++j)
              __builtin_amdgcn_raw_buffer_store_b128(*(const u32x4*)(l + j * 1024), mrs, off + (j >> 2) * s2 + ((j >> 1) & 1) * s1 + (j & 1) * 1024, 0, AUX_SYS);
          }
          WAIT_VM(0);
          {
            const unsigned got = min((unsigned)__builtin_amdgcn_readfirstlane(tk) - tbase, 0xffffu);
            if (drawing) { todo |= (unsigned long long)got << (16 * ntodo); ++ntodo; drawing = got < (unsigned)ntask; }
            tk = __builtin_amdgcn_raw_ptr_buffer_atomic_add_i32(1, mrs, drawing ? qoff : 0xfffffff0u, 0, 0);
          }
          if (t.type != 3) {
            if (lane == 0) st_sys(myws + flag, call);
            if (lane == 1 && wi != 0 && t.type == 0 && (hf ? t.a >= 32 - NF : t.a < NF))
              st_sys(p.ws[rank ^ 4] + GFLAG + (((t.a & 7) * 8 + peer) * 8 + t.b) * 4, call);
          }
          progress = true;
        }
        if (ended && kf == ka) break;
        if (progress) { polls = 0; t0 = __builtin_amdgcn_s_memrealtime(); }
        else {
          if ((polls & 63) == 0 && give_up(tmo, call, t0)) dead = true;
          __builtin_amdgcn_s_sleep(ST_SLEEP);
        }
      }
      if (lane < 4) ctl_st(4 + lane, 0x7fff8000);
      if (wi == 0) STAMPW(8 + wave);
    }
    return;
  }

#pragma unroll 1
  for (int jb = wi == 7 ? 2 : (wi < 14 ? 1 : (wi < 26 ? 0 : -1)); jb < (wi < 15 ? 3 : 2); ++jb) {
    const int job = max(jb, 0);
    const int jw = (wi - 8) / 6, jc = job == 2 ? (wi >> 1) & 3 : (jb < 0 ? 0 : jw);
    const int dj = wi - 8 - 6 * jw, peer = (rank ^ xmask(dj)) & 7;
    const int lrb = job == 2 ? wi & 1 : (job == 1 ? 2 + dj : peer >> 2);
    if (jb == 0 && jw == 2) for (int d = 0; d < MINI_DELAY; ++d) __builtin_amdgcn_s_sleep(127);
    const int cp = xcd + 8 * jc;
    const int cb = (hf && job) ? 31 - cp : cp;
    const int nst = job == 0 ? 8 : (job == 1 ? 56 : 64);
    auto stage_id = [&](int q) {
      if (q < 8) return rank * 8 + q;
      q -= 8;
      if (q >= 48) return (rank ^ 4) * 8 + q - 48;
      const int kp = q / 6, k = q - 6 * kp;
      if (job == 2) return (rank ^ xmask(k)) * 8 + kp;
      return (rank ^ (k == 0 ? 4 : xmask(k - 1 + (k > dj)))) * 8 + kp;
    };
    const int my_id = stage_id(lane);
    const int my_a = ((my_id & 7) * 8 + (my_id >> 3)) * 16384;
    const int my_b = job ? XS_BYTES + cb * (512 << 10) + (my_a >> 1) : we_off(peer & 3, cb, lane & 7);
    const unsigned* const xf = (const unsigned*)(myws + XFLAG) + lrb * 128 + (my_id >> 1) * 4;
    const unsigned* const wf = job ? (const unsigned*)(myws + WFLAG) + cb * 64 + my_id
                                   : (const unsigned*)(myws + EFLAG) + ((peer & 3) * 32 + cb) * 8 + (lane & 7);
    int ready = 0;
    auto ensure = [&](int q) {
      if (ready > q) return;
      const unsigned long long t0 = __builtin_amdgcn_s_memrealtime();
      unsigned polls = 0;
      for (;;) {
        u32x4 fx; unsigned fw;
        asm volatile("global_load_dwordx4 %0, %2, off sc0 sc1\n\t"
                     "global_load_dword %1, %3, off sc0 sc1\n\t"
                     "s_waitcnt vmcnt(0)"
                     : "=&v"(fx), "=&v"(fw) : "v"(xf), "v"(wf) : "memory");
        const unsigned long long m = __ballot(min(min4(fx), fw) >= call);
        ready = (~m == 0ull) ? 64 : __builtin_ctzll(~m);
        if (ready > q) return;
        if ((++polls & 15) == 0 && give_up(tmo, call, t0)) { ready = 64; return; }
        __builtin_amdgcn_s_sleep(POLL_SLEEP);
      }
    };
    const unsigned a_base = par + (unsigned)lrb * (1u << 20) + wave * 2048;
    const unsigned b_base = par + wave * 1024;
    auto issue = [&](int q, int slot) {
      const unsigned ao = a_base + __builtin_amdgcn_readlane(my_a, q), bo = b_base + __builtin_amdgcn_readlane(my_b, q);
      char* l = lds + slot * STAGE;
      auto* l0 = LDS3(l + wave * 2048);
      auto* l1 = LDS3(l + wave * 2048 + 1024);
      auto* l2 = LDS3(l + 16384 + wave * 1024);
      __builtin_amdgcn_raw_ptr_buffer_load_lds(mrs, l0, 16, lane * 16, ao, 0, AUX_SYS);
      __builtin_amdgcn_raw_ptr_buffer_load_lds(mrs, l1, 16, lane * 16, ao + 1024, 0, AUX_SYS);
      __builtin_amdgcn_raw_ptr_buffer_load_lds(mrs, l2, 16, lane * 16, bo, 0, AUX_SYS);
    };
    i32x16 acc[4];
#pragma unroll
    for (int b = 0; b < 4; ++b)
#pragma unroll
      for (int i = 0; i < 16; ++i) acc[b][i] = 0;
    i32x4 fa[2][2], fb[2][2][4];
    auto rd = [&](int buf, int slot) {
      const char* l = lds + slot * STAGE + lane * 16;
#pragma unroll
      for (int ks = 0; ks < 2; ++ks) {
        fa[buf][ks] = *(const i32x4*)(l + ks * 8192 + wave * 1024);
#pragma unroll
        for (int b = 0; b < 4; ++b) fb[buf][ks][b] = *(const i32x4*)(l + 16384 + ks * 4096 + b * 1024);
      }
    };
    auto mm = [&](int buf) {
#pragma unroll
      for (int ks = 0; ks < 2; ++ks)
#pragma unroll
        for (int b = 0; b < 4; ++b) acc[b] = __builtin_amdgcn_mfma_i32_32x32x32_i8(fa[buf][ks], fb[buf][ks][b], acc[b], 0, 0, 0);
    };
    auto nx = [&](int s) { return s == NSLOT - 1 ? 0 : s + 1; };

    if (wave == 0) ensure(NSLOT - 1);
    RAW_BARRIER();
    STAMP(1);
#pragma unroll
    for (int q = 0; q < NSLOT; ++q) issue(q, q);
    WAIT_VM(15);
    RAW_BARRIER();
    rd(0, 0);
    int s0 = 0;
#pragma unroll 1
    for (int i = 0; i < nst - NSLOT; i += 2) {
      if (wave == 0) ensure(i + NSLOT);
      WAIT_VM(12); RAW_BARRIER();
      issue(i + NSLOT, s0); rd(1, nx(s0)); mm(0);
      s0 = nx(s0);
      if (wave == 0) ensure(i + NSLOT + 1);
      WAIT_VM(12); RAW_BARRIER();
      issue(i + NSLOT + 1, s0); rd(0, nx(s0)); mm(1);
      s0 = nx(s0);
    }
    STAMP(2);
    WAIT_VM(12); RAW_BARRIER(); rd(1, nx(s0)); mm(0); s0 = nx(s0);
    WAIT_VM(9);  RAW_BARRIER(); rd(0, nx(s0)); mm(1); s0 = nx(s0);
    WAIT_VM(6);  RAW_BARRIER(); rd(1, nx(s0)); mm(0); s0 = nx(s0);
    WAIT_VM(3);  RAW_BARRIER(); rd(0, nx(s0)); mm(1); s0 = nx(s0);
    WAIT_VM(0);  RAW_BARRIER(); rd(1, nx(s0)); mm(0);
    mm(1);
    STAMP(3);

    const bool bad = timed_out(tmo, call);
    if (job == 2) {
      const float scale = p.sx[0] * p.sw[0];
      const unsigned poison = bad ? 0x7fc00000u : 0u;
      const unsigned ooff = (unsigned)(lrb * 256 + wave * 32 + 4 * h) * (GN * 4) + ((rank & 3) * 4096 + cb * 128) * 4 + c31 * 16;
#pragma unroll
      for (int i = 0; i < 16; ++i) {
        u32x4 o;
#pragma unroll
        for (int b = 0; b < 4; ++b) o[b] = __builtin_bit_cast(unsigned, silu_dq(acc[b][i], scale)) | poison;
        __builtin_amdgcn_raw_buffer_store_b128(o, ors, ooff + ((i & 3) + 8 * (i >> 2)) * (GN * 4), 0, OUT_AUX);
      }
    } else {
      const unsigned soff = par + (job ? OS : PS) + (unsigned)((dj * 32 + cb) * 8 + wave) * 16384 + lane * 16;
#pragma unroll
      for (int i = 0; i < 16; ++i) {
        const u32x4 o = {(unsigned)acc[0][i], (unsigned)acc[1][i], (unsigned)acc[2][i], (unsigned)acc[3][i]};
        __builtin_amdgcn_raw_buffer_store_b128(o, mrs, soff + i * 1024, 0, AUX_SYS);
      }
      WAIT_VM(0);
      RAW_BARRIER();
      if (wave == 0 && lane == 0 && !bad) st_sys(job ? p.ws[peer] + DONE + (rank * 32 + cb) * 4 : myws + PFLAG + (dj * 32 + cb) * 4, call);
    }
    STAMP(4);
  }
}

}

extern "C" void kernel_launch(int rank, int nranks, ncclComm_t comm,
                              hipStream_t stream,
                              void* const* d_in, const int* in_sizes, int n_in,
                              void* d_out, int out_size,
                              const int* in_dtypes, int out_dtype,
                              void* const* d_out_all,
                              void* const* d_ws_all, size_t ws_bytes) {
  (void)comm; (void)in_sizes; (void)n_in; (void)d_out_all; (void)out_size; (void)in_dtypes; (void)out_dtype;
  if (nranks != R || (unsigned)rank >= (unsigned)R || ws_bytes < WS_USED) {
    fprintf(stderr, "unsupported configuration\n");
    return;
  }
  Params p;
  p.x = (const int8_t*)d_in[0];
  p.w = (const int8_t*)d_in[1];
  p.sx = (const float*)d_in[2];
  p.sw = (const float*)d_in[3];
  p.out = (float*)d_out;
  for (int i = 0; i < R; ++i) p.ws[i] = (char*)d_ws_all[i];
  p.rank = rank; p.pad = 0;
  p.myws = p.ws[rank];
  fused_kernel<<<NWG, NTHR, 0, stream>>>(p);
  HIP_CHECK(hipGetLastError());
}
```

```cpp
#include <hip/hip_runtime.h>
#include <rccl/rccl.h>
#include <cstdio>
#include <cstdint>

#define HIP_CHECK(expr) do { hipError_t _e = (expr); if (_e != hipSuccess) { \
    fprintf(stderr, "HIP error %s at %s:%d\n", hipGetErrorString(_e), __FILE__, __LINE__); \
    return; } } while (0)

#define PULL_WIN 2
#define PUSH_WIN 2
#define RING_D 3
#define OUT_AUX 17
#define STG_DELAY 20
#define PUSHN 20
#define KQ_U 3
#define TAILN 8
#define MINI_DELAY 3
#define ST_SLEEP 2
#define POLL_SLEEP 4
#define NF 7
#define NFK 3
#define LG1 4

namespace {

typedef int      i32x4  __attribute__((ext_vector_type(4)));
typedef int      i32x16 __attribute__((ext_vector_type(16)));
typedef unsigned u32x4  __attribute__((ext_vector_type(4)));

constexpr int R = 8;
constexpr int GN = 16384;
constexpr int KS = 512;
constexpr int NWG = 256, NTHR = 512;

constexpr unsigned XS_BYTES = 8u << 20;
constexpr unsigned WS_BYTES = 16u << 20;
constexpr unsigned XE = XS_BYTES + WS_BYTES;
constexpr unsigned WE = XE + (2u << 20);
constexpr unsigned OS = WE + (8u << 20);
constexpr unsigned PS = OS + (24u << 20);
constexpr unsigned BUF_BYTES = PS + (24u << 20);
constexpr unsigned CTL = 2 * BUF_BYTES;
constexpr unsigned XFLAG = CTL;
constexpr unsigned WFLAG = XFLAG + 8 * 128 * 4;
constexpr unsigned DONE = WFLAG + 32 * 64 * 4;
constexpr unsigned SFLAG = DONE + 1024;
constexpr unsigned EFLAG = SFLAG + 16384;
constexpr unsigned PFLAG = EFLAG + 5120;
constexpr unsigned CNT = PFLAG + 1024;
constexpr unsigned TMO = CNT + 1024;
constexpr unsigned QUE = TMO + 1024;
constexpr unsigned WS_USED = QUE + 1024;

constexpr int NB = 16 - NF, T_0 = PUSHN  , T_A = 160, T_B = 224, T_X = T_B + 8 * NB, T_O = T_X + 32, N_OTHER = T_O + 512, N_PART = T_O + 56 * NF + 6 * NFK;
static_assert(T_0 == 20 && NF <= 7 && NFK <= 8 && RING_D <= 3, "see the staging flags / task lists");
constexpr int STAGE = 24576, NSLOT = 6;
constexpr unsigned long long DEADLINE = 50000000ull;
constexpr int AUX_SYS = 17;

struct Params {
  const int8_t* x; const int8_t* w; char* myws; int rank; int pad;
  const float* sx; const float* sw;
  float* out; char* ws[R];
};

#define WAIT_VM(n) asm volatile("s_waitcnt vmcnt(" #n ")" ::: "memory")
#define WAIT_VM_(n) WAIT_VM(n)
#define RAW_BARRIER() do { asm volatile("s_waitcnt lgkmcnt(0)" ::: "memory"); __builtin_amdgcn_s_barrier(); asm volatile("" ::: "memory"); } while (0)
#define LDS3(p) ((__attribute__((address_space(3))) void*)(p))

__device__ __forceinline__ __amdgpu_buffer_rsrc_t make_rsrc(const void* p, unsigned bytes) {
  return __builtin_amdgcn_make_buffer_rsrc((void*)p, 0, bytes, 0x00020000);
}
__device__ __forceinline__ void st_sys(void* p, unsigned v) {
  __hip_atomic_store((unsigned*)p, v, __ATOMIC_RELAXED, __HIP_MEMORY_SCOPE_SYSTEM);
}
__device__ __forceinline__ bool timed_out(unsigned* tmo, unsigned call) {
  return __builtin_amdgcn_readfirstlane(__hip_atomic_load(tmo, __ATOMIC_RELAXED, __HIP_MEMORY_SCOPE_AGENT)) == call;
}
__device__ __forceinline__ bool give_up(unsigned* tmo, unsigned call, unsigned long long t0) {
  if (timed_out(tmo, call) || __builtin_amdgcn_s_memrealtime() - t0 > DEADLINE) {
    __hip_atomic_store(tmo, call, __ATOMIC_RELAXED, __HIP_MEMORY_SCOPE_AGENT);
    return true;
  }
  return false;
}
__device__ __forceinline__ int xmask(int j) { return j + 1 + (j >= 3); }
__device__ __forceinline__ unsigned min4(u32x4 f) { return min(min(f.x, f.y), min(f.z, f.w)); }

__device__ __forceinline__ void transpose4(unsigned d0, unsigned d1, unsigned d2, unsigned d3,
                                           unsigned& o0, unsigned& o1, unsigned& o2, unsigned& o3) {
  unsigned t0 = __builtin_amdgcn_perm(d1, d0, 0x05010400u);
  unsigned t1 = __builtin_amdgcn_perm(d1, d0, 0x07030602u);
  unsigned t2 = __builtin_amdgcn_perm(d3, d2, 0x05010400u);
  unsigned t3 = __builtin_amdgcn_perm(d3, d2, 0x07030602u);
  o0 = __builtin_amdgcn_perm(t2, t0, 0x05040100u);
  o1 = __builtin_amdgcn_perm(t2, t0, 0x07060302u);
  o2 = __builtin_amdgcn_perm(t3, t1, 0x05040100u);
  o3 = __builtin_amdgcn_perm(t3, t1, 0x07060302u);
}

__device__ __forceinline__ float silu_dq(int acc, float s) {
  float y = (float)acc * s;
  float c = __builtin_amdgcn_fmed3f(y, -60.0f, 60.0f);
  float e = __builtin_amdgcn_exp2f(c * -1.44269504088896341f);
  return y * __builtin_amdgcn_rcpf(1.0f + e);
}

__device__ __forceinline__ unsigned xs_off(int lrb, int src, int b) { return (unsigned)(lrb * 64 + (b >> 2) * 16 + src) * 16384 + (b & 3) * 2048; }
__device__ __forceinline__ unsigned xs_piece(int j) { return (j >> 2) * 131072 + ((j >> 1) & 1) * 8192 + (j & 1) * 1024; }
__device__ __forceinline__ unsigned ws_off(int cb, int src, int kp) { return XS_BYTES + (unsigned)(cb * 64 + kp * 8 + src) * 8192; }
__device__ __forceinline__ unsigned xe_off(int rb, int b) { return XE + (unsigned)((b * 2 + (rb & 1)) * 8 + (rb >> 1)) * 8192; }
__device__ __forceinline__ unsigned we_off(int q, int cb, int kp) { return WE + (unsigned)((q * 8 + kp) * 32 + cb) * 8192; }

struct Task { int type, a, b, src; };

__global__ void __launch_bounds__(NTHR, 2) fused_kernel(Params p) {
  __shared__ __attribute__((aligned(16))) char lds[STAGE * NSLOT];
  const int lane = threadIdx.x & 63;
  const int wave = __builtin_amdgcn_readfirstlane(threadIdx.x >> 6);
  const int bid = blockIdx.x;
  const int rank = p.rank, hf = rank >> 2;
  char* const myws = p.myws;
  const int8_t* const px = p.x;
  const int8_t* const pw = p.w;
  asm volatile("" :: "s"(px), "s"(pw), "s"(myws), "s"(rank));
  unsigned* const tmo = (unsigned*)(myws + TMO);
  unsigned* const cnt = (unsigned*)(myws + CNT) + bid;
  const unsigned cnt_v = __hip_atomic_load(cnt, __ATOMIC_RELAXED, __HIP_MEMORY_SCOPE_AGENT);
  const int h = lane >> 5, c31 = lane & 31;
  const int xcd = bid & 7, wi = bid >> 3;
  const __amdgpu_buffer_rsrc_t mrs = make_rsrc(myws, WS_USED);
  const __amdgpu_buffer_rsrc_t ors = make_rsrc(p.out, 512u * GN * 4u);

  char* const lws = p.ws[lane & 7];
  const int n = bid - 56, sc = (wave - 1) * 200 + n;
  const int pu = n >> 3, pd = (n + pu) & 7, pm = pd ^ rank;
  const int prq = pm == 4 ? pu / 6 : pu / 5, pv = pu - (pm == 4 ? 6 : 5) * prq;
  const int plrb = 2 + pv + (pm != 4 && pv >= pm - 1 - (pm > 4));
  const bool active = n >= 0 && (wave == 0 ? (n < 8 * T_0 && pd != rank) : sc < 1280);
  int kind = 0, ca = 0, cbk = 0, ckp = 0;
  if (wave > 0) {
    const int kq = sc / 192, u = sc < 768 ? sc - kq * 192 : sc - 768;
    if (sc < 768 && u >= 128) { kind = 1; ca = u & 15; cbk = kq * 4 + ((u - 128) >> 4); }
    else {
      kind = 2; ca = (u >> 4) & 3;
      if (sc < 768) { ckp = 2 * kq + (u >> 6); cbk = (u & 15) + (u & 8) * 2; } else { ckp = u >> 6; cbk = 8 + (u & 15); }
    }
  }
  if (active) {
    const int r8 = lane >> 3, k8 = lane & 7;
    const char* src = kind == 0 ? (const char*)px + (size_t)(512 * (pd ^ xmask(plrb - 2)) + 256 * (pd >> 2) + 64 * prq) * KS
                    : kind == 1 ? (const char*)px + (size_t)(ca * 256 + (cbk & 3) * 64) * KS + (cbk >> 2) * 128
                                : (const char*)pw + (size_t)(64 * ckp) * GN + ca * 4096 + cbk * 128;
    const unsigned stride = kind < 2 ? KS : GN;
    const __amdgpu_buffer_rsrc_t srs = make_rsrc(src, 63 * stride + 128);
    const unsigned voff = r8 * stride + ((kind < 2 ? k8 ^ r8 : k8) * 16);
    for (int d = 1; d < wave; ++d) __builtin_amdgcn_s_sleep(STG_DELAY);
#pragma unroll
    for (int j = 0; j < 8; ++j)
      __builtin_amdgcn_raw_ptr_buffer_load_lds(srs, LDS3(lds + wave * 8192 + j * 1024), 16, voff, j * 8 * stride, 0, 0);
  }
  asm volatile("" ::: "memory");
  const unsigned call = __builtin_amdgcn_readfirstlane(cnt_v) + 1u;
  const unsigned par = (call & 1u) * BUF_BYTES;
  if (active) {
    WAIT_VM(0);
    const char* l = lds + wave * 8192;
    unsigned flag = 0;
    if (kind == 1) {
      const int rb = ca, b = cbk;
      auto lrb_of = [&](int d) { const int m = (rb >> 1) ^ d; return m == 0 ? (rb & 1) : (((m & 3) && (rb & 1) == (d >> 2)) ? 1 + m - (m > 4) : -1); };
      const int lrb = lrb_of(rank), lrbl = lrb_of(lane & 7);
      const unsigned goff = lrb >= 0 ? par + xs_off(lrb, rank, b) + lane * 16 : 0xf0000000u;
      if (lane == rank) { if (lrb >= 0) flag = XFLAG + ((lrb * 8 + rank) * 16 + b) * 4; }
      else if (lane == 8 + rank) flag = EFLAG + 4096 + (rb * 16 + b) * 4;
      else if (lane < 8 && lrbl >= 2 && b < 4 * KQ_U && (rb >> 1) != rank && (b >= 4 || (lane ^ rank) == 4))
        flag = SFLAG + rank * 2048 + 1024 + (lrbl * 16 + b) * 4;
#pragma unroll
      for (int j = 0; j < 8; ++j) {
        const u32x4 v = *(const u32x4*)(l + c31 * 128 + (((2 * (j >> 1) + h) ^ (c31 & 7)) * 16) + (j & 1) * 4096);
        __builtin_amdgcn_raw_buffer_store_b128(v, mrs, par + xe_off(rb, b) + j * 1024 + lane * 16, 0, AUX_SYS);
        __builtin_amdgcn_raw_buffer_store_b128(v, mrs, goff + xs_piece(j), 0, AUX_SYS);
      }
    } else if (kind == 0) {
      const __amdgpu_buffer_rsrc_t drs = make_rsrc(p.ws[pd], CTL);
      const unsigned goff = par + xs_off(plrb, rank, prq) + lane * 16;
      if (lane == pd) flag = XFLAG + ((plrb * 8 + rank) * 16 + prq) * 4;
#pragma unroll
      for (int j = 0; j < 8; ++j) {
        const u32x4 v = *(const u32x4*)(l + c31 * 128 + (((2 * (j >> 1) + h) ^ (c31 & 7)) * 16) + (j & 1) * 4096);
        __builtin_amdgcn_raw_buffer_store_b128(v, drs, goff + xs_piece(j), 0, AUX_SYS);
        WAIT_VM_(PUSH_WIN);
      }
    } else {
      const unsigned goff = ca == (rank & 3) ? par + ws_off(cbk, rank, ckp) + lane * 16 : 0xf0000000u;
      if (lane == rank) { if (ca == (rank & 3)) flag = WFLAG + ((cbk * 8 + rank) * 8 + ckp) * 4; }
      else if (lane == 8 + rank) flag = EFLAG + ((ca * 32 + cbk) * 8 + ckp) * 4;
      else if (sc < 192 * KQ_U && lane == ca + (cbk >= 24 ? 4 : 0)) flag = SFLAG + rank * 2048 + (cbk * 8 + ckp) * 4;
#pragma unroll
      for (int kk = 0; kk < 2; ++kk) {
        unsigned o[4][4];
#pragma unroll
        for (int q = 0; q < 4; ++q) {
          const unsigned* lp = (const unsigned*)(l + (kk * 32 + h * 16 + q * 4) * 128 + c31 * 4);
          transpose4(lp[0], lp[32], lp[64], lp[96], o[0][q], o[1][q], o[2][q], o[3][q]);
        }
#pragma unroll
        for (int b = 0; b < 4; ++b) {
          const u32x4 v = {o[b][0], o[b][1], o[b][2], o[b][3]};
          __builtin_amdgcn_raw_buffer_store_b128(v, mrs, par + we_off(ca, cbk, ckp) + (kk * 4 + b) * 1024 + lane * 16, 0, AUX_SYS);
          __builtin_amdgcn_raw_buffer_store_b128(v, mrs, goff + (kk * 4 + b) * 1024, 0, AUX_SYS);
        }
      }
    }
    WAIT_VM(0);
    if (flag) st_sys(lws + flag, call);
  }
  const unsigned ctl = (unsigned)(size_t)(__attribute__((address_space(3))) char*)(lds + 131072 + (wave & 1) * 32);
  auto ctl_st = [&](int slot, int v) { asm volatile("ds_write_b32 %0, %1" :: "v"(ctl + slot * 4), "v"(v) : "memory"); };
  auto ctl_ld = [&](int slot) {
    int v; asm volatile("ds_read_b32 %0, %1\n\ts_waitcnt lgkmcnt(0)" : "=&v"(v) : "v"(ctl + slot * 4) : "memory");
    return __builtin_amdgcn_readfirstlane(v);
  };
  if (wave >= 6 && lane < 8) ctl_st(lane, lane < 4 ? 0 : -1);
  asm volatile("" :: "s"(call));
  RAW_BARRIER();
  if (wave == 7 && lane == 0) __hip_atomic_store(cnt, call, __ATOMIC_RELAXED, __HIP_MEMORY_SCOPE_AGENT);

  if (wi < 7) {
    const int pair = wave & 1;
    const int peer = wi == 0 ? rank ^ 4 : rank ^ xmask(wi - 1);
    const int ntask = wi == 0 ? N_PART : N_OTHER;
    constexpr int cn = 16;
    const int sh = wi == 0 ? 0 : NFK;
    const int i0 = (wi == 0 ? 0 : 16) + sh + T_0;
    const int cw = i0 + pair * 8 + xcd;
    char* const pws = p.ws[peer];
    char* const ring = lds + pair * 32768;
    auto decode = [&](int i) {
      Task t; t.src = peer;
      int cp = 0;
      if (i < T_A + sh) {
        int kq, u;
        if (wi == 0) {
          kq = i / 40; u = i - kq * 40;
          if (u < 24) { const int rq = u / 6; t.type = 1; t.a = 2 + u - 6 * rq; t.b = kq * 4 + rq; return t; }
          u -= 24;
        } else {
          kq = (i - 16 - NFK) / 36; u = i - 16 - NFK - kq * 36;
          if (u < 20) { const int rq = u / 5, v = u - 5 * rq; t.type = 1; t.a = 2 + v + (v >= wi - 1); t.b = kq * 4 + rq; return t; }
          u -= 20;
        }
        t.type = 0; cp = u & 7; t.b = 2 * kq + (u >> 3);
      } else if (i < T_B + sh) { t.type = 0; cp = 8 + ((i - sh) & 7); t.b = (i - sh - T_A) >> 3;
      } else if (i < T_X - 8 + sh) { t.type = 0; t.b = (i - sh - T_B) / (NB - 1); cp = 16 + i - sh - T_B - (NB - 1) * t.b;
      } else if (i < T_X) { t.type = 0; cp = 31 - NF; t.b = i - (T_X - 8); }
      else if (wi != 0 ? i < T_O : i >= N_PART - 32) {
        const int u = i - (wi != 0 ? T_X : N_PART - 32);
        t.type = 1; t.a = u & 1; t.b = u >> 1; return t;
      } else if (wi != 0) {
        t.type = 3; t.b = (i - T_O) & 15; cp = (i - T_O) >> 4; t.a = (peer >> 2) ? 31 - cp : cp; return t;
      } else if (i < T_X + 6 * NFK) {
        t.type = 2; cp = 31 - NF; t.b = (i - T_X) / 6; t.src = rank ^ xmask(i - T_X - 6 * t.b);
      } else {
        int j = i - T_X - 6 * NFK, kp, u;
        if (j < LG1 * 56) { kp = j / (LG1 * 7); u = j - kp * (LG1 * 7); }
        else { j -= LG1 * 56; kp = j / ((NF - LG1) * 7); u = j - kp * ((NF - LG1) * 7) + LG1 * 7; }
        const int jb = u / 7, k = u - jb * 7;
        cp = 32 - NF + jb; t.b = kp;
        if (k == 0) t.type = 0; else { t.type = 2; t.src = rank ^ xmask(k - 1); }
      }
      t.a = hf ? 31 - cp : cp;
      return t;
    };
    if (wave & 2) {
      if (wi == 0) {
        if (wave != 2 || bid > 1) return;
        const bool skip = bid == 1 && (((lane & 3) == (rank & 3)) || (hf ? (lane >> 3) < 7 - NF : (lane >> 3) > NF));
        const unsigned base = bid == 0 ? EFLAG + lane * 80 : WFLAG + (hf ? 24 * 256 : 0) + lane * 32;
        const unsigned long long t0 = __builtin_amdgcn_s_memrealtime();
        for (unsigned polls = 1;; ++polls) {
          unsigned m = ~0u;
#pragma unroll
          for (int j = 0; j < 5; ++j)
            m = min(m, min4(__builtin_amdgcn_raw_buffer_load_b128(mrs, base + (bid == 0 ? j : j & 1) * 16, 0, AUX_SYS)));
          asm volatile("" ::: "memory");
          if (__all(skip || m >= call)) break;
          if ((polls & 15) == 0 && give_up(tmo, call, t0)) return;
          __builtin_amdgcn_s_sleep(4);
        }
        if (timed_out(tmo, call)) return;
        if (bid == 0 ? (lane < 8 && lane != rank) : lane == (rank ^ 4)) st_sys(lws + SFLAG + rank * 2048 + 1536 + bid * 4, call);
        return;
      }
      const float scale = p.sx[0] * p.sw[0];
#pragma unroll 1
      for (int k = wave >> 2;; k += 2) {
        const int slot = k & 3;
        int a;
        while (((a = ctl_ld(4 + slot)) >> 16) < k) __builtin_amdgcn_s_sleep(4);
        if (a & 0x8000) break;
        if ((a >> 16) > k || (a & 3) != 3) continue;
        const int cb = (a >> 2) & 31, part = (a >> 7) & 15;
        const unsigned poff = par + PS + (unsigned)(((wi - 1) * 32 + cb) * 16 + part) * 8192 + lane * 16;
        u32x4 v[8];
#pragma unroll
        for (int j = 0; j < 8; ++j) v[j] = __builtin_amdgcn_raw_buffer_load_b128(mrs, poff + j * 1024, 0, AUX_SYS);
        while (ctl_ld(slot) != -1) __builtin_amdgcn_s_sleep(2);
#pragma unroll
        for (int j = 0; j < 8; ++j) v[j] += *(const u32x4*)(ring + slot * 8192 + lane * 16 + j * 1024);
        asm volatile("" : "+v"(v[0]), "+v"(v[1]), "+v"(v[2]), "+v"(v[3]), "+v"(v[4]), "+v"(v[5]), "+v"(v[6]), "+v"(v[7]) :: "memory");
        ctl_st(slot, -3);
        const unsigned off = (unsigned)((peer >> 2) * 256 + part * 16 + 4 * h) * (GN * 4) + ((peer & 3) * 4096 + cb * 128) * 4 + c31 * 16;
        const unsigned poison = (a & 0x4000) ? 0x7fc00000u : 0u;
#pragma unroll
        for (int j = 0; j < 8; ++j) {
          u32x4 o;
#pragma unroll
          for (int b = 0; b < 4; ++b) o[b] = __builtin_bit_cast(unsigned, silu_dq((int)v[j][b], scale)) | poison;
          __builtin_amdgcn_raw_buffer_store_b128(o, ors, off + ((j & 3) + 8 * (j >> 2)) * (GN * 4), 0, OUT_AUX);
        }
      }
    } else if (wave >= 4) {
      const __amdgpu_buffer_rsrc_t prs = make_rsrc(pws, CTL);
      int pend = -1;
#pragma unroll 1
      for (int k = 0;; ++k) {
        const int slot = k & 3;
        int c = ctl_ld(slot);
        if (c <= 0 && c != -2) {
          if (pend >= 0) { WAIT_VM(0); ctl_st(pend, -1); pend = -1; }
          while ((c = ctl_ld(slot)) <= 0 && c != -2) __builtin_amdgcn_s_sleep(1);
        }
        if (c == -2) break;
        const unsigned soff = (unsigned)(c >> 1) << 13;
        if (c & 1) {
#pragma unroll
          for (int j = 0; j < 8; ++j)
            __builtin_amdgcn_raw_ptr_buffer_load_lds(prs, LDS3(ring + slot * 8192 + j * 1024), 16, lane * 16, soff + j * 1024, 0, AUX_SYS);
          WAIT_VM(8);
          if (pend >= 0) ctl_st(pend, -1);
        } else {
#pragma unroll
          for (int j = 0; j < 8; ++j) {
            __builtin_amdgcn_raw_ptr_buffer_load_lds(prs, LDS3(ring + slot * 8192 + j * 1024), 16, lane * 16, soff + j * 1024, 0, AUX_SYS);
            WAIT_VM_(PULL_WIN);
            if (j == (PULL_WIN > 0 ? PULL_WIN - 1 : 0) && pend >= 0) ctl_st(pend, -1);
          }
        }
        pend = slot;
      }
      if (pend >= 0) { WAIT_VM(0); ctl_st(pend, -1); }
    } else {
      const unsigned qoff = lane == 0 ? QUE + peer * 128 : 0xfffffff0u;
      const unsigned tbase = (call - 1u) * (unsigned)(ntask - i0 - (RING_D - 1) * cn) - (unsigned)(i0 + RING_D * cn);
      bool staged = false, given = false, ended = false, drawing = true, dead = false;
      unsigned tiles = 0;
      u32x4 sf0 = {0, 0, 0, 0}, sf1 = {0, 0, 0, 0};
      unsigned long long todo = 0;
      int ntodo = RING_D;
#pragma unroll
      for (int j = 0; j < RING_D; ++j) todo |= (unsigned long long)(cw + j * cn) << (16 * j);
      unsigned long long held = 0;
      int ka = 0, kf = 0;
      int tk = __builtin_amdgcn_raw_ptr_buffer_atomic_add_i32(1, mrs, qoff, 0, 0);
      unsigned long long t0 = __builtin_amdgcn_s_memrealtime();
      auto ready = [&](unsigned i) {
        if (dead) return true;
        if (!staged || (wi == 0 && i >= T_X && !given)) {
          const bool urgent = i < (wi == 0 ? 40 * KQ_U : 16 + NFK + 36 * KQ_U);
          const Task t = decode(urgent ? (int)i : i0);
          const int w = t.type == 0 ? t.a * 8 + t.b : t.a * 16 + t.b;
          for (int pass = 0;; ++pass) {
            const u32x4 f = t.type == 0 ? sf0 : sf1;
            const unsigned fw = (w & 2) ? ((w & 1) ? f.w : f.z) : ((w & 1) ? f.y : f.x);
            if (urgent && (unsigned)__builtin_amdgcn_readlane(fw, w >> 2) >= call) return true;
            if (pass) return false;
            asm volatile("global_load_dwordx4 %0, %2, off sc0 sc1\n\t"
                         "global_load_dwordx4 %1, %2, off offset:1024 sc0 sc1\n\t"
                         "s_waitcnt vmcnt(0)" : "=&v"(sf0), "=&v"(sf1) : "v"(myws + SFLAG + peer * 2048 + lane * 16) : "memory");
            staged = (unsigned)__builtin_amdgcn_readlane(sf1.x, 32) >= call;
            given = (unsigned)__builtin_amdgcn_readlane(sf1.y, 32) >= call;
            if (staged && (wi != 0 || i < T_X || given)) break;
          }
        }
        if (wi == 0) return true;
        if (i < T_O) return true;
        const unsigned cp = (i - T_O) >> 4, bit = 1u << ((peer >> 2) ? 31 - cp : cp);
        if (tiles & bit) return true;
        unsigned f;
        asm volatile("global_load_dword %0, %1, off sc0 sc1\n\ts_waitcnt vmcnt(0)" : "=&v"(f)
                     : "v"(myws + (h ? PFLAG + (wi - 1) * 128 : DONE + peer * 128) + c31 * 4) : "memory");
        const unsigned long long m = __ballot(f >= call);
        tiles = (unsigned)m & (unsigned)(m >> 32);
        return (tiles & bit) != 0;
      };
      auto assign = [&]() {
        bool any = false;
        while (ntodo > 0) {
          const unsigned nxt = (unsigned)todo & 0xffffu;
          if (nxt >= (unsigned)ntask) { ctl_st(ka & 3, -2); ended = true; ntodo = 0; break; }
          if (!ready(nxt)) break;
          {
            const Task t = decode((int)nxt);
            unsigned soff;
            if (t.type == 0) soff = we_off(rank & 3, t.a, t.b);
            else if (t.type == 1) soff = xe_off(t.a < 2 ? 2 * rank + t.a : 2 * (rank ^ xmask(t.a - 2)) + hf, t.b);
            else if (t.type == 2) soff = ws_off(t.a, t.src, t.b);
            else soff = OS + (unsigned)(((wi - 1) * 32 + t.a) * 16 + t.b) * 8192;
            const unsigned pk = t.type | (t.a << 2) | (t.b << 7) | (t.src << 11) | (dead ? 0x4000 : 0);
            ctl_st(ka & 3, (int)(((par + soff) >> 12) | (nxt + TAILN >= (unsigned)ntask ? 1u : 0u)));
            ctl_st(4 + (ka & 3), (int)((ka << 16) | pk));
            held = (held & ~(0xffffull << (16 * (ka & 3)))) | ((unsigned long long)pk << (16 * (ka & 3)));
          }
          ++ka; any = true;
          todo >>= 16; --ntodo;
        }
        return any;
      };
#pragma unroll 1
      for (unsigned polls = 1;; ++polls) {
        bool progress = assign();
        const int pk = (int)(held >> (16 * (kf & 3)));
        if (kf < ka && ctl_ld(kf & 3) == ((pk & 3) == 3 ? -3 : -1)) {
          const char* l = ring + (kf & 3) * 8192 + lane * 16;
          const Task t = {pk & 3, (pk >> 2) & 31, (pk >> 7) & 15, (pk >> 11) & 7};
          ++kf;
          unsigned flag = 0;
          if (t.type != 3) {
            unsigned off, s1, s2;
            if (t.type == 1) {
              off = xs_off(t.a, peer, t.b); s1 = 8192; s2 = 131072;
              flag = XFLAG + ((t.a * 8 + peer) * 16 + t.b) * 4;
            } else {
              off = ws_off(t.a, t.src, t.b); s1 = 2048; s2 = 4096;
              flag = WFLAG + ((t.a * 8 + t.src) * 8 + t.b) * 4;
            }
            off += par + lane * 16;
#pragma unroll
            for (int j = 0; j < 8; ++j)
              __builtin_amdgcn_raw_buffer_store_b128(*(const u32x4*)(l + j * 1024), mrs, off + (j >> 2) * s2 + ((j >> 1) & 1) * s1 + (j & 1) * 1024, 0, AUX_SYS);
          }
          WAIT_VM(0);
          {
            const unsigned got = min((unsigned)__builtin_amdgcn_readfirstlane(tk) - tbase, 0xffffu);
            if (drawing) { todo |= (unsigned long long)got << (16 * ntodo); ++ntodo; drawing = got < (unsigned)ntask; }
            tk = __builtin_amdgcn_raw_ptr_buffer_atomic_add_i32(1, mrs, drawing ? qoff : 0xfffffff0u, 0, 0);
          }
          if (t.type != 3 && lane == 0) st_sys(myws + flag, call);
          progress = true;
        }
        if (ended && kf == ka) break;
        if (progress) { polls = 0; t0 = __builtin_amdgcn_s_memrealtime(); }
        else {
          if ((polls & 63) == 0 && give_up(tmo, call, t0)) dead = true;
          __builtin_amdgcn_s_sleep(ST_SLEEP);
        }
      }
      if (lane < 4) ctl_st(4 + lane, 0x7fff8000);
    }
    return;
  }

#pragma unroll 1
  for (int jb = wi == 7 ? 2 : (wi < 14 ? 1 : (wi < 26 ? 0 : -1)); jb < (wi < 15 ? 3 : 2); ++jb) {
    const int job = max(jb, 0);
    const int jw = (wi - 8) / 6, jc = job == 2 ? (wi >> 1) & 3 : (jb < 0 ? 0 : jw);
    const int dj = wi - 8 - 6 * jw, peer = (rank ^ xmask(dj)) & 7;
    const int lrb = job == 2 ? wi & 1 : (job == 1 ? 2 + dj : peer >> 2);
    if (jb == 0 && jw == 2) for (int d = 0; d < MINI_DELAY; ++d) __builtin_amdgcn_s_sleep(127);
    const int cp = xcd + 8 * jc;
    const int cb = (hf && job) ? 31 - cp : cp;
    const int nst = job == 0 ? 8 : (job == 1 ? 56 : 64);
    auto stage_id = [&](int q) {
      if (q < 8) return rank * 8 + q;
      q -= 8;
      if (q >= 48) return (rank ^ 4) * 8 + q - 48;
      const int kp = q / 6, k = q - 6 * kp;
      if (job == 2) return (rank ^ xmask(k)) * 8 + kp;
      return (rank ^ (k == 0 ? 4 : xmask(k - 1 + (k > dj)))) * 8 + kp;
    };
    const int my_id = stage_id(lane);
    const int my_a = ((my_id & 7) * 8 + (my_id >> 3)) * 16384;
    const int my_b = job ? XS_BYTES + cb * (512 << 10) + (my_a >> 1) : we_off(peer & 3, cb, lane & 7);
    const unsigned* const xf = (const unsigned*)(myws + XFLAG) + lrb * 128 + (my_id >> 1) * 4;
    const unsigned* const wf = job ? (const unsigned*)(myws + WFLAG) + cb * 64 + my_id
                                   : (const unsigned*)(myws + EFLAG) + ((peer & 3) * 32 + cb) * 8 + (lane & 7);
    int ready = 0;
    auto ensure = [&](int q) {
      if (ready > q) return;
      const unsigned long long t0 = __builtin_amdgcn_s_memrealtime();
      unsigned polls = 0;
      for (;;) {
        u32x4 fx; unsigned fw;
        asm volatile("global_load_dwordx4 %0, %2, off sc0 sc1\n\t"
                     "global_load_dword %1, %3, off sc0 sc1\n\t"
                     "s_waitcnt vmcnt(0)"
                     : "=&v"(fx), "=&v"(fw) : "v"(xf), "v"(wf) : "memory");
        const unsigned long long m = __ballot(min(min4(fx), fw) >= call);
        ready = (~m == 0ull) ? 64 : __builtin_ctzll(~m);
        if (ready > q) return;
        if ((++polls & 15) == 0 && give_up(tmo, call, t0)) { ready = 64; return; }
        __builtin_amdgcn_s_sleep(POLL_SLEEP);
      }
    };
    const unsigned a_base = par + (unsigned)lrb * (1u << 20) + wave * 2048;
    const unsigned b_base = par + wave * 1024;
    auto issue = [&](int q, int slot) {
      const unsigned ao = a_base + __builtin_amdgcn_readlane(my_a, q), bo = b_base + __builtin_amdgcn_readlane(my_b, q);
      char* l = lds + slot * STAGE;
      auto* l0 = LDS3(l + wave * 2048);
      auto* l1 = LDS3(l + wave * 2048 + 1024);
      auto* l2 = LDS3(l + 16384 + wave * 1024);
      __builtin_amdgcn_raw_ptr_buffer_load_lds(mrs, l0, 16, lane * 16, ao, 0, AUX_SYS);
      __builtin_amdgcn_raw_ptr_buffer_load_lds(mrs, l1, 16, lane * 16, ao + 1024, 0, AUX_SYS);
      __builtin_amdgcn_raw_ptr_buffer_load_lds(mrs, l2, 16, lane * 16, bo, 0, AUX_SYS);
    };
    i32x16 acc[4];
#pragma unroll
    for (int b = 0; b < 4; ++b)
#pragma unroll
      for (int i = 0; i < 16; ++i) acc[b][i] = 0;
    i32x4 fa[2][2], fb[2][2][4];
    auto rd = [&](int buf, int slot) {
      const char* l = lds + slot * STAGE + lane * 16;
#pragma unroll
      for (int ks = 0; ks < 2; ++ks) {
        fa[buf][ks] = *(const i32x4*)(l + ks * 8192 + wave * 1024);
#pragma unroll
        for (int b = 0; b < 4; ++b) fb[buf][ks][b] = *(const i32x4*)(l + 16384 + ks * 4096 + b * 1024);
      }
    };
    auto mm = [&](int buf) {
#pragma unroll
      for (int ks = 0; ks < 2; ++ks)
#pragma unroll
        for (int b = 0; b < 4; ++b) acc[b] = __builtin_amdgcn_mfma_i32_32x32x32_i8(fa[buf][ks], fb[buf][ks][b], acc[b], 0, 0, 0);
    };
    auto nx = [&](int s) { return s == NSLOT - 1 ? 0 : s + 1; };

    if (wave == 0) ensure(NSLOT - 1);
    RAW_BARRIER();
#pragma unroll
    for (int q = 0; q < NSLOT; ++q) issue(q, q);
    WAIT_VM(15);
    RAW_BARRIER();
    rd(0, 0);
    int s0 = 0;
#pragma unroll 1
    for (int i = 0; i < nst - NSLOT; i += 2) {
      if (wave == 0) ensure(i + NSLOT);
      WAIT_VM(12); RAW_BARRIER();
      issue(i + NSLOT, s0); rd(1, nx(s0)); mm(0);
      s0 = nx(s0);
      if (wave == 0) ensure(i + NSLOT + 1);
      WAIT_VM(12); RAW_BARRIER();
      issue(i + NSLOT + 1, s0); rd(0, nx(s0)); mm(1);
      s0 = nx(s0);
    }
    WAIT_VM(12); RAW_BARRIER(); rd(1, nx(s0)); mm(0); s0 = nx(s0);
    WAIT_VM(9);  RAW_BARRIER(); rd(0, nx(s0)); mm(1); s0 = nx(s0);
    WAIT_VM(6);  RAW_BARRIER(); rd(1, nx(s0)); mm(0); s0 = nx(s0);
    WAIT_VM(3);  RAW_BARRIER(); rd(0, nx(s0)); mm(1); s0 = nx(s0);
    WAIT_VM(0);  RAW_BARRIER(); rd(1, nx(s0)); mm(0);
    mm(1);

    const bool bad = timed_out(tmo, call);
    if (job == 2) {
      const float scale = p.sx[0] * p.sw[0];
      const unsigned poison = bad ? 0x7fc00000u : 0u;
      const unsigned ooff = (unsigned)(lrb * 256 + wave * 32 + 4 * h) * (GN * 4) + ((rank & 3) * 4096 + cb * 128) * 4 + c31 * 16;
#pragma unroll
      for (int i = 0; i < 16; ++i) {
        u32x4 o;
#pragma unroll
        for (int b = 0; b < 4; ++b) o[b] = __builtin_bit_cast(unsigned, silu_dq(acc[b][i], scale)) | poison;
        __builtin_amdgcn_raw_buffer_store_b128(o, ors, ooff + ((i & 3) + 8 * (i >> 2)) * (GN * 4), 0, OUT_AUX);
      }
    } else {
      const unsigned soff = par + (job ? OS : PS) + (unsigned)((dj * 32 + cb) * 8 + wave) * 16384 + lane * 16;
#pragma unroll
      for (int i = 0; i < 16; ++i) {
        const u32x4 o = {(unsigned)acc[0][i], (unsigned)acc[1][i], (unsigned)acc[2][i], (unsigned)acc[3][i]};
        __builtin_amdgcn_raw_buffer_store_b128(o, mrs, soff + i * 1024, 0, AUX_SYS);
      }
      WAIT_VM(0);
      RAW_BARRIER();
      if (wave == 0 && lane == 0 && !bad) st_sys(job ? p.ws[peer] + DONE + (rank * 32 + cb) * 4 : myws + PFLAG + (dj * 32 + cb) * 4, call);
    }
  }
}

}

extern "C" void kernel_launch(int rank, int nranks, ncclComm_t comm,
                              hipStream_t stream,
                              void* const* d_in, const int* in_sizes, int n_in,
                              void* d_out, int out_size,
                              const int* in_dtypes, int out_dtype,
                              void* const* d_out_all,
                              void* const* d_ws_all, size_t ws_bytes) {
  (void)comm; (void)in_sizes; (void)n_in; (void)d_out_all; (void)out_size; (void)in_dtypes; (void)out_dtype;
  if (nranks != R || (unsigned)rank >= (unsigned)R || ws_bytes < WS_USED) {
    fprintf(stderr, "unsupported configuration\n");
    return;
  }
  Params p;
  p.x = (const int8_t*)d_in[0];
  p.w = (const int8_t*)d_in[1];
  p.sx = (const float*)d_in[2];
  p.sw = (const float*)d_in[3];
  p.out = (float*)d_out;
  for (int i = 0; i < R; ++i) p.ws[i] = (char*)d_ws_all[i];
  p.rank = rank; p.pad = 0;
  p.myws = p.ws[rank];
  fused_kernel<<<NWG, NTHR, 0, stream>>>(p);
  HIP_CHECK(hipGetLastError());
}
```
